```python
import numpy as np

kernel_name = "gpu_a2a_gemm_m4096_k4096_n16384_bf16_none"


ABSMAX_RTOL = 0.008

COMPUTE_DTYPE = "bfloat16"
EPILOGUE = "none"
IN_QUANT = None
OUT_QUANT = None
IN_DTYPES = {"x": "bfloat16", "w_mat": "bfloat16"}
WORLD_SIZE = 8

def _to_bf16(x):
    u = np.ascontiguousarray(x, dtype=np.float32).view(np.uint32)
    return ((u + 0x7FFF + ((u >> 16) & 1)) & 0xFFFF0000).view(np.float32)

def setup_inputs():

    x = _to_bf16(np.random.randn(4096, 4096).astype(np.float32))
    w_mat = _to_bf16((np.random.randn(4096, 16384) / np.sqrt(4096)).astype(np.float32))
    return {"x": x, "w_mat": w_mat}

def _epilogue(y):
    return y

def reference(x, w_mat):
    y = (x @ w_mat).astype(np.float32)
    return _epilogue(y)
```

```cpp
#include <hip/hip_runtime.h>
#include <rccl/rccl.h>
#include <cstdio>

#define HIP_CHECK(expr) do { hipError_t _e = (expr); if (_e != hipSuccess) { \
    fprintf(stderr, "HIP error %s at %s:%d\n", hipGetErrorString(_e), __FILE__, __LINE__); \
    return; } } while (0)

#define LAS __attribute__((address_space(3)))
typedef __bf16 bf16x8 __attribute__((ext_vector_type(8)));
typedef __bf16 bf16x2 __attribute__((ext_vector_type(2)));
typedef short s16x4 __attribute__((ext_vector_type(4)));
typedef short s16x8 __attribute__((ext_vector_type(8)));
typedef float f32x4 __attribute__((ext_vector_type(4)));
typedef float f32x16 __attribute__((ext_vector_type(16)));
typedef float f32x2 __attribute__((ext_vector_type(2)));
typedef unsigned u32x4 __attribute__((ext_vector_type(4)));
typedef unsigned u32x2 __attribute__((ext_vector_type(2)));
typedef int i32x4 __attribute__((ext_vector_type(4)));

#define LOADER_SLEEP 1
#define SPREAD 0
#define AUX_A 0
#define AUX_B 0
#define WIN 2
#define ST_AUX 17
#define PUSH_T0 150
#define PUSH_DT 35
#define POLL_SLEEP 4
#define ACQ 0
#define MF32 0

namespace {

constexpr int NR = 8;
constexpr int MLOC = 512;
constexpr int KCH = 512;
constexpr int NTOT = 16384;
constexpr int NSLOT = 6;
constexpr int A_SLOT = 8192;
constexpr int B_SLOT = 16384;
constexpr int SLOT = A_SLOT + B_SLOT;
constexpr int CTL_OFF = NSLOT * SLOT;
constexpr int LDS_BYTES = CTL_OFF + 64;
constexpr int NSTEP = 128;
constexpr int NWG = 256;

constexpr unsigned FLAG_OFF = 0;
constexpr unsigned TMO_OFF = 2048;
constexpr unsigned CTR_OFF = 4096;
constexpr unsigned DATA_OFF = 65536;
constexpr unsigned PAR_BYTES = 4u << 20;
constexpr unsigned WS_NEED = DATA_OFF + 2 * PAR_BYTES;
constexpr unsigned long long DEADLINE = 50000000ull;

struct Params {
    const void* x;
    const void* w;
    void* out;
    char* ws[NR];
    int rank;
};

__device__ __forceinline__ __amdgpu_buffer_rsrc_t mk_rsrc(const void* p, unsigned bytes) {
    return __builtin_amdgcn_make_buffer_rsrc((void*)p, (short)0, (int)bytes, 0x00020000);
}

#define DMA(...) __builtin_amdgcn_raw_ptr_buffer_load_lds(__VA_ARGS__)
#define DMA_A(...) DMA(__VA_ARGS__)
#define DMA_B(...) DMA(__VA_ARGS__)
#define WAIT_VM(n) asm volatile("s_waitcnt vmcnt(" #n ")" ::: "memory")
#define LAG 3
#define WAIT_LAG() WAIT_VM(24)
#define WAIT_LGKM0() asm volatile("s_waitcnt lgkmcnt(0)" ::: "memory")

__device__ __forceinline__ unsigned pack2(float a, float b) {
    f32x2 v = {a, b};
    bf16x2 r = __builtin_convertvector(v, bf16x2);
    return __builtin_bit_cast(unsigned, r);
}

__global__ __launch_bounds__(512, 2) void a2a_gemm_kernel(Params p) {
    extern __shared__ __attribute__((aligned(16))) char smem[];
    LAS char* L = (LAS char*)smem;
    volatile LAS unsigned* ctl = (volatile LAS unsigned*)(L + CTL_OFF);

    const int tid = threadIdx.x;
    const int wave = __builtin_amdgcn_readfirstlane(tid >> 6);
    const int lane = tid & 63;
    const int b = blockIdx.x;
    const int idx = b >> 3;
    const int nb = (b & 7) * 8 + (idx >> 2);
    const int mb = idx & 3;
    const int rank = p.rank;

#define STAMP(i) do {} while (0)
    STAMP(0);
#define TR(J, i) do {} while (0)
    if (tid < 16) ctl[tid] = 0u;
    __syncthreads();

    if (wave < 4) {
        const int wn = wave;
        u32x4 fl;
        auto wait_full = [&](unsigned need) {
            for (;;) {
                u32x4 f = *(volatile LAS u32x4*)(L + CTL_OFF);
                unsigned mn = min(min(f[0], f[1]), f[2]);
                if (__builtin_amdgcn_readfirstlane(mn) >= need) break;
            }
            asm volatile("" ::: "memory");
        };
        bf16x8 xa[2][8], wb[2][4];
        const int g = lane >> 4, r15 = lane & 15, q = (lane >> 2) & 3, pp = lane & 3;
        const unsigned aoff = (unsigned)(r15 * 64 + ((g ^ ((-(r15 >> 2)) & 3)) * 16));
        unsigned boff[4];
#pragma unroll
        for (int t = 0; t < 4; ++t)
            boff[t] = (unsigned)(A_SLOT + (8 * g + q) * 512 + (((((wn & 1) * 4 + t) ^ (q | ((g & 1) << 2)))) << 5) +
                                 ((wn >> 1) << 8) + 8 * pp);
        f32x4 acc[8][4];
#pragma unroll
        for (int m = 0; m < 8; ++m)
#pragma unroll
            for (int t = 0; t < 4; ++t) acc[m][t] = (f32x4){0.f, 0.f, 0.f, 0.f};
        auto rd_w = [&](unsigned sb, int t) -> bf16x8 {
            s16x4 lo = __builtin_amdgcn_ds_read_tr16_b64_v4i16((LAS s16x4*)(L + sb + boff[t]));
            s16x4 hi = __builtin_amdgcn_ds_read_tr16_b64_v4i16((LAS s16x4*)(L + sb + boff[t] + 2048));
            s16x8 v = __builtin_shufflevector(lo, hi, 0, 1, 2, 3, 4, 5, 6, 7);
            return __builtin_bit_cast(bf16x8, v);
        };
        auto rd_x = [&](unsigned sb, int m) -> bf16x8 {
            return *(const LAS bf16x8*)(L + sb + aoff + m * 1024);
        };
#define SB_() __builtin_amdgcn_sched_barrier(0)
#define READ_ALL(U, SBASE, PIN) do { \
            wb[U][0] = rd_w(SBASE, 0); PIN; \
            _Pragma("unroll") for (int m = 0; m < 8; ++m) { xa[U][m] = rd_x(SBASE, m); PIN; } \
            _Pragma("unroll") for (int t = 1; t < 4; ++t) { wb[U][t] = rd_w(SBASE, t); PIN; } \
            fl = *(volatile LAS u32x4*)(L + CTL_OFF); PIN; } while (0)
#define MMA(U, I) acc[(I) & 7][(I) >> 3] = __builtin_amdgcn_mfma_f32_16x16x32_bf16(wb[U][(I) >> 3], xa[U][(I) & 7], acc[(I) & 7][(I) >> 3], 0, 0, 0)
#define NTAIL16 5
        constexpr int NMMA = 32, NTAIL = NTAIL16;

        wait_full(1);
        STAMP(1);
        READ_ALL(0, 0u, SB_());
        WAIT_LGKM0();
        ctl[4 + wn] = 1u;
        wait_full(2);

        unsigned sbn = SLOT;
        for (int J = 0; J < NSTEP; J += 2) {
#pragma unroll
            for (int u = 0; u < 2; ++u) {
                const int Jc = J + u;
                READ_ALL(u ^ 1, sbn, (void)0);
#pragma unroll
                for (int i = 0; i < NMMA - NTAIL; ++i) MMA(u, i);
                __builtin_amdgcn_sched_group_barrier(0x008, 1, 0);
#pragma unroll
                for (int i = 0; i < 17; ++i) {
                    __builtin_amdgcn_sched_group_barrier(0x100, 1, 0);
                    __builtin_amdgcn_sched_group_barrier(0x008, 1, 0);
                }
                __builtin_amdgcn_sched_group_barrier(0x008, NMMA - NTAIL - 18, 0);
                __builtin_amdgcn_sched_barrier(0);
                WAIT_LGKM0();
                TR(Jc, 0);
                ctl[4 + wn] = (unsigned)(Jc + 2);
                sbn += SLOT;
                if (sbn == NSLOT * SLOT) sbn = 0;
                asm volatile("" : "+s"(sbn));
                const unsigned mn = __builtin_amdgcn_readfirstlane(min(min(fl[0], fl[1]), fl[2]));
                const unsigned need = (unsigned)min(Jc + 3, NSTEP);
#pragma unroll
                for (int i = NMMA - NTAIL; i < NMMA; ++i) MMA(u, i);
                __builtin_amdgcn_sched_group_barrier(0x002, 1, 0);
                __builtin_amdgcn_sched_group_barrier(0x008, 1, 0);
                __builtin_amdgcn_sched_group_barrier(0x002, 1, 0);
                __builtin_amdgcn_sched_group_barrier(0x008, 1, 0);
                __builtin_amdgcn_sched_group_barrier(0x002, 2, 0);
                __builtin_amdgcn_sched_group_barrier(0x200, 1, 0);
                __builtin_amdgcn_sched_group_barrier(0x008, NTAIL - 2, 0);
                if (__builtin_expect(mn < need, 0)) wait_full(need);
                TR(Jc, 1);
                asm volatile("" ::: "memory");
            }
        }

        WAIT_LGKM0();
        STAMP(2);
        const unsigned bad = ctl[8];
        const __amdgpu_buffer_rsrc_t ro = mk_rsrc(p.out, (unsigned)MLOC * NTOT * 2u);
        const unsigned obase = (unsigned)(((mb * 128 + r15) * NTOT + nb * 256 + wn * 64 + 16 * (g & 1) + 8 * (g >> 1)) * 2);
#pragma unroll
        for (int m = 0; m < 8; ++m) {
#pragma unroll
            for (int tp = 0; tp < 2; ++tp) {
                unsigned a0 = pack2(acc[m][2 * tp][0], acc[m][2 * tp][1]);
                unsigned a1 = pack2(acc[m][2 * tp][2], acc[m][2 * tp][3]);
                unsigned b0 = pack2(acc[m][2 * tp + 1][0], acc[m][2 * tp + 1][1]);
                unsigned b1 = pack2(acc[m][2 * tp + 1][2], acc[m][2 * tp + 1][3]);
                auto s0 = __builtin_amdgcn_permlane16_swap(a0, b0, false, false);
                auto s1 = __builtin_amdgcn_permlane16_swap(a1, b1, false, false);
                u32x4 v = {s0[0], s1[0], s0[1], s1[1]};
                if (bad) v = (u32x4){0x7FC07FC0u, 0x7FC07FC0u, 0x7FC07FC0u, 0x7FC07FC0u};
                __builtin_amdgcn_raw_buffer_store_b128(v, ro, (int)obase, (m * 16 * NTOT + tp * 32) * 2, 0);
            }
        }
    } else if (wave < 7) {
        const int ld = wave - 4;
        volatile LAS int* fullw = (volatile LAS int*)(L + CTL_OFF) + ld;
        i32x4 epre = {0, 0, 0, 0};
        int pub = 0;
        int J = 0;
        unsigned sb = 0;
        auto rd_freed = [&]() -> int {
            i32x4 e = *(volatile LAS i32x4*)(L + CTL_OFF + 16);
            return __builtin_amdgcn_readfirstlane(min(min(e[0], e[1]), min(e[2], e[3])));
        };
        auto pre = [&]() {
            const int need = J - (NSLOT - 1);
            int freed = __builtin_amdgcn_readfirstlane(min(min(epre[0], epre[1]), min(epre[2], epre[3])));
            if (__builtin_expect(freed < need, 0)) {
                if (pub < J - 1) { WAIT_VM(8); pub = J - 1; *fullw = pub; }
                freed = rd_freed();
                if (freed < need) {
                    if (pub < J) { WAIT_VM(0); pub = J; *fullw = pub; }
                    while ((freed = rd_freed()) < need) __builtin_amdgcn_s_sleep(LOADER_SLEEP);
                }
            }
            asm volatile("" ::: "memory");
        };
        auto post = [&]() {
            epre = *(volatile LAS i32x4*)(L + CTL_OFF + 16);
            WAIT_VM(16);
            pub = max(pub, J - 1);
            *fullw = pub;
            ++J;
            sb += SLOT;
            if (sb == NSLOT * SLOT) sb = 0;
        };
        if (ld == 0) {
            const __amdgpu_buffer_rsrc_t rx = mk_rsrc(p.x, 4096u * KCH * 2u);
            char* wsme = p.ws[rank];
            const __amdgpu_buffer_rsrc_t rw = mk_rsrc(wsme, WS_NEED);
            const unsigned callv = __hip_atomic_load((const unsigned*)(wsme + CTR_OFF + b * 128), __ATOMIC_RELAXED, __HIP_MEMORY_SCOPE_AGENT);
            const unsigned vloc = (unsigned)((lane >> 2) * 1024 + (((lane & 3) ^ ((-(lane >> 4)) & 3)) * 16));
            const unsigned vrem = (unsigned)(lane * 16);
            const unsigned call = __builtin_amdgcn_readfirstlane(callv) + 1u;
            unsigned so = (unsigned)((rank * 512 + mb * 128) * 1024);
#pragma unroll 1
            for (int j = 0; j < 16; ++j) {
                pre();
#pragma unroll
                for (int i = 0; i < 8; ++i)
                    DMA_A(rx, (LAS void*)(L + sb + i * 1024), 16, (int)vloc, (int)(so + i * 16 * 1024), 0, AUX_A);
                post();
                so += 64;
            }
            const unsigned dbase = DATA_OFF + (call & 1u) * PAR_BYTES + (unsigned)(mb * 16 * 8192);
            unsigned long long okm = 0ull;
            int s7 = 0, jj = 0;
#pragma unroll 1
            for (int t = 0; t < 56; ++t) {
                const int src = (rank + 1 + s7) & 7;
                const int bit = src * 8 + jj;
                if (__builtin_expect(((okm >> bit) & 1ull) == 0ull, 0)) {
                    auto rd_mask = [&]() -> unsigned long long {
                        u32x2 m2 = *(volatile LAS u32x2*)(L + CTL_OFF + 40);
                        return ((unsigned long long)(unsigned)__builtin_amdgcn_readfirstlane(m2[1]) << 32) |
                               (unsigned)__builtin_amdgcn_readfirstlane(m2[0]);
                    };
                    okm = rd_mask();
                    if (((okm >> bit) & 1ull) == 0ull) {
                        if (pub < J) { WAIT_VM(0); pub = J; *fullw = pub; }
                        while ((((okm = rd_mask()) >> bit) & 1ull) == 0ull) __builtin_amdgcn_s_sleep(1);
                    }
                    asm volatile("" ::: "memory");
                }
                const unsigned sr = dbase + (unsigned)((src * 64 + 2 * jj) * 8192);
#pragma unroll
                for (int hf = 0; hf < 2; ++hf) {
                    pre();
#pragma unroll
                    for (int i = 0; i < 8; ++i)
                        DMA_A(rw, (LAS void*)(L + sb + i * 1024), 16, (int)vrem, (int)(sr + hf * 8192 + i * 1024), 0, AUX_A);
                    post();
                }
                ++s7;
                if (s7 == 7) { s7 = 0; ++jj; }
            }
        } else {
            const int bl = ld - 1;
            const __amdgpu_buffer_rsrc_t rwm = mk_rsrc(p.w, 4096u * NTOT * 2u);
            const int h = lane >> 5;
            unsigned vo[4];
#pragma unroll
            for (int c = 0; c < 4; ++c)
                vo[c] = (unsigned)(h * 32768 + (((lane & 31) ^ ((2 * c + h) << 1)) * 16));
            const unsigned wbase = (unsigned)(16 * bl * 32768 + nb * 512);
            const unsigned lo = (unsigned)(A_SLOT + bl * 8192);
            auto issue = [&](unsigned so) {
#pragma unroll
                for (int i = 0; i < 8; ++i)
                    DMA_B(rwm, (LAS void*)(L + sb + lo + i * 1024), 16, (int)vo[(i & 1) + 2 * ((i >> 2) & 1)],
                          (int)(so + i * 2 * 32768), 0, AUX_B);
            };
            unsigned so = wbase + (unsigned)rank * (512u * 32768u);
#pragma unroll 1
            for (int j = 0; j < 16; ++j) {
                pre();
                issue(so);
                post();
                so += 32u * 32768u;
            }
            int s7 = 0, jj = 0;
#pragma unroll 1
            for (int t = 0; t < 56; ++t) {
                const int src = (rank + 1 + s7) & 7;
                const unsigned sr = wbase + (unsigned)(src * 512 + jj * 64) * 32768u;
#pragma unroll
                for (int hf = 0; hf < 2; ++hf) {
                    pre();
                    issue(sr + hf * 32u * 32768u);
                    post();
                }
                ++s7;
                if (s7 == 7) { s7 = 0; ++jj; }
            }
        }
        WAIT_VM(8);  *fullw = NSTEP - 1;
        WAIT_VM(0);  *fullw = NSTEP;
    } else {
        const unsigned long long tk0 = __builtin_amdgcn_s_memrealtime();
        char* wsme = p.ws[rank];
        unsigned* ctr = (unsigned*)(wsme + CTR_OFF + b * 128);
        const unsigned callv = __hip_atomic_load(ctr, __ATOMIC_RELAXED, __HIP_MEMORY_SCOPE_AGENT);
        const int pi = b % 7, un = b / 7, jjp = (un >> 2) & 7, mbd = un & 3;
        const int dst = (rank + 1 + pi) & 7;
        char* wsd = p.ws[dst];
        const __amdgpu_buffer_rsrc_t rx = mk_rsrc(p.x, 4096u * KCH * 2u);
        const __amdgpu_buffer_rsrc_t rd = mk_rsrc(wsd, WS_NEED);
        const unsigned vs = (unsigned)((lane >> 2) * 1024 + (((lane & 3) ^ ((-(lane >> 4)) & 3)) * 16));
        const unsigned ss = (unsigned)((dst * 512 + mbd * 128) * 1024 + jjp * 128);
        u32x4 v[16];
        bool todo = b < 224;
        if (todo) {
#pragma unroll
            for (int j = 0; j < 16; ++j)
                v[j] = __builtin_amdgcn_raw_buffer_load_b128(rx, (int)vs, (int)(ss + (j & 7) * 16384 + (j >> 3) * 64), 0);
        }
        __builtin_amdgcn_sched_barrier(0);
        WAIT_VM(0);
        const unsigned call = __builtin_amdgcn_readfirstlane(callv) + 1u;
        const unsigned long long due = tk0 + (unsigned long long)(PUSH_T0 + un * PUSH_DT);
        const unsigned* fptr = (const unsigned*)(wsme + FLAG_OFF + (lane >> 3) * 128 + (mb * 8 + (lane & 7)) * 4);
        unsigned* tmo = (unsigned*)(wsme + TMO_OFF);
        const unsigned long long want = ~(0xFFull << (rank * 8));
        unsigned long long have = 0ull;
        for (;;) {
            if (todo && __builtin_amdgcn_s_memrealtime() >= due) {
                const unsigned sd = DATA_OFF + (call & 1u) * PAR_BYTES + (unsigned)(((rank * 4 + mbd) * 16 + 2 * jjp) * 8192);
                unsigned* flag = (unsigned*)(wsd + FLAG_OFF + rank * 128 + mbd * 32 + jjp * 4);
#pragma unroll
                for (int j = 0; j < 16; ++j)
                    __builtin_amdgcn_raw_buffer_store_b128(v[j], rd, lane * 16, (int)(sd + j * 1024), ST_AUX);
                WAIT_VM(0);
                if (lane == 0) __hip_atomic_store(flag, call, __ATOMIC_RELAXED, __HIP_MEMORY_SCOPE_SYSTEM);
                todo = false;
            }
            if (have != want) {
                const unsigned fv = __hip_atomic_load(fptr, __ATOMIC_RELAXED, __HIP_MEMORY_SCOPE_SYSTEM);
                unsigned long long now = __ballot(fv >= call) & want;
                const unsigned long long el = __builtin_amdgcn_s_memrealtime() - tk0;
                if (now != want && el > 100000ull &&
                    (el > DEADLINE ||
                     __builtin_amdgcn_readfirstlane(__hip_atomic_load(tmo, __ATOMIC_RELAXED, __HIP_MEMORY_SCOPE_AGENT)) == call)) {
                    __hip_atomic_store(tmo, call, __ATOMIC_RELAXED, __HIP_MEMORY_SCOPE_AGENT);
                    ctl[8] = 1u;
                    now = want;
                }
                if (now != have) {
                    have = now;
                    *(volatile LAS u32x2*)(L + CTL_OFF + 40) = (u32x2){(unsigned)have, (unsigned)(have >> 32)};
                }
            } else if (!todo) break;
            __builtin_amdgcn_s_sleep(POLL_SLEEP);
        }
        for (;;) {
            u32x4 e = *(volatile LAS u32x4*)(L + CTL_OFF + 16);
            if (__builtin_amdgcn_readfirstlane(e[0]) >= 1u) break;
            __builtin_amdgcn_s_sleep(8);
        }
        if (lane == 0) __hip_atomic_store(ctr, call, __ATOMIC_RELAXED, __HIP_MEMORY_SCOPE_AGENT);
    }
}

}

extern "C" void kernel_launch(int rank, int nranks, ncclComm_t comm,
                              hipStream_t stream,
                              void* const* d_in, const int* in_sizes, int n_in,
                              void* d_out, int out_size,
                              const int* in_dtypes, int out_dtype,
                              void* const* d_out_all,
                              void* const* d_ws_all, size_t ws_bytes) {
    (void)comm; (void)in_sizes; (void)n_in; (void)out_size; (void)in_dtypes; (void)out_dtype; (void)d_out_all;
    if (nranks != NR || (unsigned)rank >= (unsigned)NR || ws_bytes < WS_NEED) {
        fprintf(stderr, "unexpected configuration\n");
        return;
    }
    Params p;
    p.x = d_in[0];
    p.w = d_in[1];
    p.out = d_out;
    for (int i = 0; i < NR; ++i) p.ws[i] = (char*)d_ws_all[i];
    p.rank = rank;
    HIP_CHECK(hipFuncSetAttribute((const void*)a2a_gemm_kernel, hipFuncAttributeMaxDynamicSharedMemorySize, LDS_BYTES));
    hipLaunchKernelGGL(a2a_gemm_kernel, dim3(NWG), dim3(512), LDS_BYTES, stream, p);
    HIP_CHECK(hipGetLastError());
}
```

```cpp
#include <hip/hip_runtime.h>
#include <rccl/rccl.h>
#include <cstdio>

#define HIP_CHECK(expr) do { hipError_t _e = (expr); if (_e != hipSuccess) { \
    fprintf(stderr, "HIP error %s at %s:%d\n", hipGetErrorString(_e), __FILE__, __LINE__); \
    return; } } while (0)

#define LAS __attribute__((address_space(3)))
typedef __bf16 bf16x8 __attribute__((ext_vector_type(8)));
typedef __bf16 bf16x2 __attribute__((ext_vector_type(2)));
typedef short s16x4 __attribute__((ext_vector_type(4)));
typedef short s16x8 __attribute__((ext_vector_type(8)));
typedef float f32x4 __attribute__((ext_vector_type(4)));
typedef float f32x16 __attribute__((ext_vector_type(16)));
typedef float f32x2 __attribute__((ext_vector_type(2)));
typedef unsigned u32x4 __attribute__((ext_vector_type(4)));
typedef unsigned u32x2 __attribute__((ext_vector_type(2)));
typedef int i32x4 __attribute__((ext_vector_type(4)));

#define LOADER_SLEEP 1
#define SPREAD 0
#define AUX_A 0
#define AUX_B 0
#define WIN 2
#define ST_AUX 17
#define PUSH_T0 150
#define PUSH_DT 35
#define POLL_SLEEP 4
#define ACQ 0
#define QAHEAD 4
#define MF32 0

namespace {

constexpr int NR = 8;
constexpr int MLOC = 512;
constexpr int KCH = 512;
constexpr int NTOT = 16384;
constexpr int NSLOT = 6;
constexpr int A_SLOT = 8192;
constexpr int B_SLOT = 16384;
constexpr int SLOT = A_SLOT + B_SLOT;
constexpr int CTL_OFF = NSLOT * SLOT;
constexpr int LDS_BYTES = CTL_OFF + 128;
constexpr int NSTEP = 128;
constexpr int NWG = 256;

constexpr unsigned FLAG_OFF = 0;
constexpr unsigned TMO_OFF = 2048;
constexpr unsigned CTR_OFF = 4096;
constexpr unsigned DATA_OFF = 65536;
constexpr unsigned PAR_BYTES = 4u << 20;
constexpr unsigned WS_NEED = DATA_OFF + 2 * PAR_BYTES;
constexpr unsigned long long DEADLINE = 50000000ull;

struct Params {
    const void* x;
    const void* w;
    void* out;
    char* ws[NR];
    int rank;
};

__device__ __forceinline__ __amdgpu_buffer_rsrc_t mk_rsrc(const void* p, unsigned bytes) {
    return __builtin_amdgcn_make_buffer_rsrc((void*)p, (short)0, (int)bytes, 0x00020000);
}

#define DMA(...) __builtin_amdgcn_raw_ptr_buffer_load_lds(__VA_ARGS__)
#define DMA_A(...) DMA(__VA_ARGS__)
#define DMA_B(...) DMA(__VA_ARGS__)
#define WAIT_VM(n) asm volatile("s_waitcnt vmcnt(" #n ")" ::: "memory")
#define LAG 3
#define WAIT_LAG() WAIT_VM(24)
#define WAIT_LGKM0() asm volatile("s_waitcnt lgkmcnt(0)" ::: "memory")

__device__ __forceinline__ unsigned pack2(float a, float b) {
    f32x2 v = {a, b};
    bf16x2 r = __builtin_convertvector(v, bf16x2);
    return __builtin_bit_cast(unsigned, r);
}

__global__ __launch_bounds__(512, 2) void a2a_gemm_kernel(Params p) {
    extern __shared__ __attribute__((aligned(16))) char smem[];
    LAS char* L = (LAS char*)smem;
    volatile LAS unsigned* ctl = (volatile LAS unsigned*)(L + CTL_OFF);

    const int tid = threadIdx.x;
    const int wave = __builtin_amdgcn_readfirstlane(tid >> 6);
    const int lane = tid & 63;
    const int b = blockIdx.x;
    const int idx = b >> 3;
    const int nb = (b & 7) * 8 + (idx >> 2);
    const int mb = idx & 3;
    const int rank = p.rank;

#define STAMP(i) do {} while (0)
    STAMP(0);
#define TR(J, i) do {} while (0)
    if (tid < 32) ctl[tid] = 0u;
    __syncthreads();

    if (wave < 4) {
        const int wn = wave;
        u32x4 fl;
        auto wait_full = [&](unsigned need) {
            for (;;) {
                u32x4 f = *(volatile LAS u32x4*)(L + CTL_OFF);
                unsigned mn = min(min(f[0], f[1]), f[2]);
                if (__builtin_amdgcn_readfirstlane(mn) >= need) break;
            }
            asm volatile("" ::: "memory");
        };
        bf16x8 xa[2][8], wb[2][4];
        const int g = lane >> 4, r15 = lane & 15, q = (lane >> 2) & 3, pp = lane & 3;
        const unsigned aoff = (unsigned)(r15 * 64 + ((g ^ ((-(r15 >> 2)) & 3)) * 16));
        unsigned boff[4];
#pragma unroll
        for (int t = 0; t < 4; ++t)
            boff[t] = (unsigned)(A_SLOT + (8 * g + q) * 512 + (((((wn & 1) * 4 + t) ^ (q | ((g & 1) << 2)))) << 5) +
                                 ((wn >> 1) << 8) + 8 * pp);
        f32x4 acc[8][4];
#pragma unroll
        for (int m = 0; m < 8; ++m)
#pragma unroll
            for (int t = 0; t < 4; ++t) acc[m][t] = (f32x4){0.f, 0.f, 0.f, 0.f};
        auto rd_w = [&](unsigned sb, int t) -> bf16x8 {
            s16x4 lo = __builtin_amdgcn_ds_read_tr16_b64_v4i16((LAS s16x4*)(L + sb + boff[t]));
            s16x4 hi = __builtin_amdgcn_ds_read_tr16_b64_v4i16((LAS s16x4*)(L + sb + boff[t] + 2048));
            s16x8 v = __builtin_shufflevector(lo, hi, 0, 1, 2, 3, 4, 5, 6, 7);
            return __builtin_bit_cast(bf16x8, v);
        };
        auto rd_x = [&](unsigned sb, int m) -> bf16x8 {
            return *(const LAS bf16x8*)(L + sb + aoff + m * 1024);
        };
#define SB_() __builtin_amdgcn_sched_barrier(0)
#define READ_ALL(U, SBASE, PIN) do { \
            wb[U][0] = rd_w(SBASE, 0); PIN; \
            _Pragma("unroll") for (int m = 0; m < 8; ++m) { xa[U][m] = rd_x(SBASE, m); PIN; } \
            _Pragma("unroll") for (int t = 1; t < 4; ++t) { wb[U][t] = rd_w(SBASE, t); PIN; } \
            fl = *(volatile LAS u32x4*)(L + CTL_OFF); PIN; } while (0)
#define MMA(U, I) acc[(I) & 7][(I) >> 3] = __builtin_amdgcn_mfma_f32_16x16x32_bf16(wb[U][(I) >> 3], xa[U][(I) & 7], acc[(I) & 7][(I) >> 3], 0, 0, 0)
#define NTAIL16 5
        constexpr int NMMA = 32, NTAIL = NTAIL16;

        wait_full(1);
        STAMP(1);
        READ_ALL(0, 0u, SB_());
        WAIT_LGKM0();
        ctl[4 + wn] = 1u;
        wait_full(2);

        unsigned sbn = SLOT;
        for (int J = 0; J < NSTEP; J += 2) {
#pragma unroll
            for (int u = 0; u < 2; ++u) {
                const int Jc = J + u;
                READ_ALL(u ^ 1, sbn, (void)0);
#pragma unroll
                for (int i = 0; i < NMMA - NTAIL; ++i) MMA(u, i);
                __builtin_amdgcn_sched_group_barrier(0x008, 1, 0);
#pragma unroll
                for (int i = 0; i < 17; ++i) {
                    __builtin_amdgcn_sched_group_barrier(0x100, 1, 0);
                    __builtin_amdgcn_sched_group_barrier(0x008, 1, 0);
                }
                __builtin_amdgcn_sched_group_barrier(0x008, NMMA - NTAIL - 18, 0);
                __builtin_amdgcn_sched_barrier(0);
                WAIT_LGKM0();
                TR(Jc, 0);
                ctl[4 + wn] = (unsigned)(Jc + 2);
                sbn += SLOT;
                if (sbn == NSLOT * SLOT) sbn = 0;
                asm volatile("" : "+s"(sbn));
                const unsigned mn = __builtin_amdgcn_readfirstlane(min(min(fl[0], fl[1]), fl[2]));
                const unsigned need = (unsigned)min(Jc + 3, NSTEP);
#pragma unroll
                for (int i = NMMA - NTAIL; i < NMMA; ++i) MMA(u, i);
                __builtin_amdgcn_sched_group_barrier(0x002, 1, 0);
                __builtin_amdgcn_sched_group_barrier(0x008, 1, 0);
                __builtin_amdgcn_sched_group_barrier(0x002, 1, 0);
                __builtin_amdgcn_sched_group_barrier(0x008, 1, 0);
                __builtin_amdgcn_sched_group_barrier(0x002, 2, 0);
                __builtin_amdgcn_sched_group_barrier(0x200, 1, 0);
                __builtin_amdgcn_sched_group_barrier(0x008, NTAIL - 2, 0);
                if (__builtin_expect(mn < need, 0)) wait_full(need);
                TR(Jc, 1);
                asm volatile("" ::: "memory");
            }
        }

        WAIT_LGKM0();
        STAMP(2);
        const unsigned bad = ctl[8];
        const __amdgpu_buffer_rsrc_t ro = mk_rsrc(p.out, (unsigned)MLOC * NTOT * 2u);
        const unsigned obase = (unsigned)(((mb * 128 + r15) * NTOT + nb * 256 + wn * 64 + 16 * (g & 1) + 8 * (g >> 1)) * 2);
#pragma unroll
        for (int m = 0; m < 8; ++m) {
#pragma unroll
            for (int tp = 0; tp < 2; ++tp) {
                unsigned a0 = pack2(acc[m][2 * tp][0], acc[m][2 * tp][1]);
                unsigned a1 = pack2(acc[m][2 * tp][2], acc[m][2 * tp][3]);
                unsigned b0 = pack2(acc[m][2 * tp + 1][0], acc[m][2 * tp + 1][1]);
                unsigned b1 = pack2(acc[m][2 * tp + 1][2], acc[m][2 * tp + 1][3]);
                auto s0 = __builtin_amdgcn_permlane16_swap(a0, b0, false, false);
                auto s1 = __builtin_amdgcn_permlane16_swap(a1, b1, false, false);
                u32x4 v = {s0[0], s1[0], s0[1], s1[1]};
                if (bad) v = (u32x4){0x7FC07FC0u, 0x7FC07FC0u, 0x7FC07FC0u, 0x7FC07FC0u};
                __builtin_amdgcn_raw_buffer_store_b128(v, ro, (int)obase, (m * 16 * NTOT + tp * 32) * 2, 0);
            }
        }
    } else if (wave < 7) {
        const int ld = wave - 4;
        volatile LAS int* fullw = (volatile LAS int*)(L + CTL_OFF) + ld;
        i32x4 epre = {0, 0, 0, 0};
        int pub = 0;
        int J = 0;
        unsigned sb = 0;
        auto rd_freed = [&]() -> int {
            i32x4 e = *(volatile LAS i32x4*)(L + CTL_OFF + 16);
            return __builtin_amdgcn_readfirstlane(min(min(e[0], e[1]), min(e[2], e[3])));
        };
        auto pre = [&]() {
            const int need = J - (NSLOT - 1);
            int freed = __builtin_amdgcn_readfirstlane(min(min(epre[0], epre[1]), min(epre[2], epre[3])));
            if (__builtin_expect(freed < need, 0)) {
                if (pub < J - 1) { WAIT_VM(8); pub = J - 1; *fullw = pub; }
                freed = rd_freed();
                if (freed < need) {
                    if (pub < J) { WAIT_VM(0); pub = J; *fullw = pub; }
                    while ((freed = rd_freed()) < need) __builtin_amdgcn_s_sleep(LOADER_SLEEP);
                }
            }
            asm volatile("" ::: "memory");
        };
        auto post = [&]() {
            epre = *(volatile LAS i32x4*)(L + CTL_OFF + 16);
            WAIT_VM(16);
            pub = max(pub, J - 1);
            *fullw = pub;
            ++J;
            sb += SLOT;
            if (sb == NSLOT * SLOT) sb = 0;
        };
        unsigned qpre = 0;
        auto q_prefetch = [&](int t) { qpre = *(volatile LAS unsigned char*)(L + CTL_OFF + 64 + t); };
        auto q_get = [&](int t) -> unsigned {
            unsigned c = __builtin_amdgcn_readfirstlane(qpre);
            if (__builtin_expect(c == 0u, 0)) {
                if (pub < J) { WAIT_VM(0); pub = J; *fullw = pub; }
                while ((c = __builtin_amdgcn_readfirstlane((unsigned)*(volatile LAS unsigned char*)(L + CTL_OFF + 64 + t))) == 0u)
                    __builtin_amdgcn_s_sleep(1);
            }
            asm volatile("" ::: "memory");
            return c;
        };
        if (ld == 0) {
            const __amdgpu_buffer_rsrc_t rx = mk_rsrc(p.x, 4096u * KCH * 2u);
            char* wsme = p.ws[rank];
            const __amdgpu_buffer_rsrc_t rw = mk_rsrc(wsme, WS_NEED);
            const unsigned callv = __hip_atomic_load((const unsigned*)(wsme + CTR_OFF + b * 128), __ATOMIC_RELAXED, __HIP_MEMORY_SCOPE_AGENT);
            const unsigned vloc = (unsigned)((lane >> 2) * 1024 + (((lane & 3) ^ ((-(lane >> 4)) & 3)) * 16));
            const unsigned vrem = (unsigned)(lane * 16);
            const unsigned call = __builtin_amdgcn_readfirstlane(callv) + 1u;
            unsigned so = (unsigned)((rank * 512 + mb * 128) * 1024);
#pragma unroll 1
            for (int j = 0; j < 16; ++j) {
                pre();
#pragma unroll
                for (int i = 0; i < 8; ++i)
                    DMA_A(rx, (LAS void*)(L + sb + i * 1024), 16, (int)vloc, (int)(so + i * 16 * 1024), 0, AUX_A);
                post();
                so += 64;
            }
            const unsigned dbase = DATA_OFF + (call & 1u) * PAR_BYTES + (unsigned)(mb * 16 * 8192);
            q_prefetch(0);
#pragma unroll 1
            for (int t = 0; t < 56; ++t) {
                const unsigned c = q_get(t);
                const unsigned sr = dbase + ((c >> 3) & 7u) * (64u * 8192u) + (c & 7u) * 16384u;
#pragma unroll
                for (int hf = 0; hf < 2; ++hf) {
                    pre();
#pragma unroll
                    for (int i = 0; i < 8; ++i)
                        DMA_A(rw, (LAS void*)(L + sb + i * 1024), 16, (int)vrem, (int)(sr + hf * 8192 + i * 1024), 0, AUX_A);
                    if (hf == 1) q_prefetch(t + 1);
                    post();
                }
            }
        } else {
            const int bl = ld - 1;
            const __amdgpu_buffer_rsrc_t rwm = mk_rsrc(p.w, 4096u * NTOT * 2u);
            const int h = lane >> 5;
            unsigned vo[4];
#pragma unroll
            for (int c = 0; c < 4; ++c)
                vo[c] = (unsigned)(h * 32768 + (((lane & 31) ^ ((2 * c + h) << 1)) * 16));
            const unsigned wbase = (unsigned)(16 * bl * 32768 + nb * 512);
            const unsigned lo = (unsigned)(A_SLOT + bl * 8192);
            auto issue = [&](unsigned so) {
#pragma unroll
                for (int i = 0; i < 8; ++i)
                    DMA_B(rwm, (LAS void*)(L + sb + lo + i * 1024), 16, (int)vo[(i & 1) + 2 * ((i >> 2) & 1)],
                          (int)(so + i * 2 * 32768), 0, AUX_B);
            };
            unsigned so = wbase + (unsigned)rank * (512u * 32768u);
#pragma unroll 1
            for (int j = 0; j < 16; ++j) {
                pre();
                issue(so);
                post();
                so += 32u * 32768u;
            }
            q_prefetch(0);
#pragma unroll 1
            for (int t = 0; t < 56; ++t) {
                const unsigned c = q_get(t);
                const unsigned sr = wbase + (((c >> 3) & 7u) * 512u + (c & 7u) * 64u) * 32768u;
#pragma unroll
                for (int hf = 0; hf < 2; ++hf) {
                    pre();
                    issue(sr + hf * 32u * 32768u);
                    if (hf == 1) q_prefetch(t + 1);
                    post();
                }
            }
        }
        WAIT_VM(8);  *fullw = NSTEP - 1;
        WAIT_VM(0);  *fullw = NSTEP;
    } else {
        const unsigned long long tk0 = __builtin_amdgcn_s_memrealtime();
        char* wsme = p.ws[rank];
        unsigned* ctr = (unsigned*)(wsme + CTR_OFF + b * 128);
        const unsigned callv = __hip_atomic_load(ctr, __ATOMIC_RELAXED, __HIP_MEMORY_SCOPE_AGENT);
        const int pi = b % 7, un = b / 7, jjp = (un >> 2) & 7, mbd = un & 3;
        const int dst = (rank + 1 + pi) & 7;
        char* wsd = p.ws[dst];
        const __amdgpu_buffer_rsrc_t rx = mk_rsrc(p.x, 4096u * KCH * 2u);
        const __amdgpu_buffer_rsrc_t rd = mk_rsrc(wsd, WS_NEED);
        const unsigned vs = (unsigned)((lane >> 2) * 1024 + (((lane & 3) ^ ((-(lane >> 4)) & 3)) * 16));
        const unsigned ss = (unsigned)((dst * 512 + mbd * 128) * 1024 + jjp * 128);
        u32x4 v[16];
        bool todo = b < 224;
        if (todo) {
#pragma unroll
            for (int j = 0; j < 16; ++j)
                v[j] = __builtin_amdgcn_raw_buffer_load_b128(rx, (int)vs, (int)(ss + (j & 7) * 16384 + (j >> 3) * 64), 0);
        }
        __builtin_amdgcn_sched_barrier(0);
        WAIT_VM(0);
        const unsigned call = __builtin_amdgcn_readfirstlane(callv) + 1u;
        const unsigned long long due = tk0 + (unsigned long long)(PUSH_T0 + un * PUSH_DT);
        const int cl = lane < 56 ? lane : 0;
        const int srcl = (rank + 1 + cl % 7) & 7, jjl = cl / 7;
        const unsigned codel = 0x80u | (unsigned)(srcl << 3) | (unsigned)jjl;
        const unsigned* fptr = (const unsigned*)(wsme + FLAG_OFF + srcl * 128 + mb * 32 + jjl * 4);
        unsigned* tmo = (unsigned*)(wsme + TMO_OFF);
        const unsigned long long want = (1ull << 56) - 1ull;
        unsigned long long have = 0ull, disp = 0ull;
        int qn = 0;
        for (;;) {
            if (todo && __builtin_amdgcn_s_memrealtime() >= due) {
                const unsigned sd = DATA_OFF + (call & 1u) * PAR_BYTES + (unsigned)(((rank * 4 + mbd) * 16 + 2 * jjp) * 8192);
                unsigned* flag = (unsigned*)(wsd + FLAG_OFF + rank * 128 + mbd * 32 + jjp * 4);
#pragma unroll
                for (int j = 0; j < 16; ++j)
                    __builtin_amdgcn_raw_buffer_store_b128(v[j], rd, lane * 16, (int)(sd + j * 1024), ST_AUX);
                WAIT_VM(0);
                if (lane == 0) __hip_atomic_store(flag, call, __ATOMIC_RELAXED, __HIP_MEMORY_SCOPE_SYSTEM);
                todo = false;
            }
            if (have != want) {
                const unsigned fv = __hip_atomic_load(fptr, __ATOMIC_RELAXED, __HIP_MEMORY_SCOPE_SYSTEM);
                unsigned long long now = __ballot(fv >= call) & want;
                const unsigned long long el = __builtin_amdgcn_s_memrealtime() - tk0;
                if (now != want && el > 100000ull &&
                    (el > DEADLINE ||
                     __builtin_amdgcn_readfirstlane(__hip_atomic_load(tmo, __ATOMIC_RELAXED, __HIP_MEMORY_SCOPE_AGENT)) == call)) {
                    __hip_atomic_store(tmo, call, __ATOMIC_RELAXED, __HIP_MEMORY_SCOPE_AGENT);
                    ctl[8] = 1u;
                    now = want;
                }
                have = now;
            }
            {
                int limit = 56;
                if (have != want) limit = min(56, max(0, ((int)__builtin_amdgcn_readfirstlane(ctl[0]) - 16) >> 1) + QAHEAD);
                unsigned long long avail = have & ~disp;
                while (qn < limit && avail) {
                    const int c = __builtin_ctzll(avail);
                    avail &= avail - 1ull;
                    disp |= 1ull << c;
                    *(volatile LAS unsigned char*)(L + CTL_OFF + 64 + qn) = (unsigned char)__builtin_amdgcn_readlane(codel, c);
                    ++qn;
                }
            }
            if (qn == 56 && !todo) break;
            __builtin_amdgcn_s_sleep(POLL_SLEEP);
        }
        for (;;) {
            u32x4 e = *(volatile LAS u32x4*)(L + CTL_OFF + 16);
            if (__builtin_amdgcn_readfirstlane(e[0]) >= 1u) break;
            __builtin_amdgcn_s_sleep(8);
        }
        if (lane == 0) __hip_atomic_store(ctr, call, __ATOMIC_RELAXED, __HIP_MEMORY_SCOPE_AGENT);
    }
}

}

extern "C" void kernel_launch(int rank, int nranks, ncclComm_t comm,
                              hipStream_t stream,
                              void* const* d_in, const int* in_sizes, int n_in,
                              void* d_out, int out_size,
                              const int* in_dtypes, int out_dtype,
                              void* const* d_out_all,
                              void* const* d_ws_all, size_t ws_bytes) {
    (void)comm; (void)in_sizes; (void)n_in; (void)out_size; (void)in_dtypes; (void)out_dtype; (void)d_out_all;
    if (nranks != NR || (unsigned)rank >= (unsigned)NR || ws_bytes < WS_NEED) {
        fprintf(stderr, "unexpected configuration\n");
        return;
    }
    Params p;
    p.x = d_in[0];
    p.w = d_in[1];
    p.out = d_out;
    for (int i = 0; i < NR; ++i) p.ws[i] = (char*)d_ws_all[i];
    p.rank = rank;
    HIP_CHECK(hipFuncSetAttribute((const void*)a2a_gemm_kernel, hipFuncAttributeMaxDynamicSharedMemorySize, LDS_BYTES));
    hipLaunchKernelGGL(a2a_gemm_kernel, dim3(NWG), dim3(512), LDS_BYTES, stream, p);
    HIP_CHECK(hipGetLastError());
}
```

```cpp
#include <hip/hip_runtime.h>
#include <rccl/rccl.h>
#include <cstdio>

#define HIP_CHECK(expr) do { hipError_t _e = (expr); if (_e != hipSuccess) { \
    fprintf(stderr, "HIP error %s at %s:%d\n", hipGetErrorString(_e), __FILE__, __LINE__); \
    return; } } while (0)

#define LAS __attribute__((address_space(3)))
typedef __bf16 bf16x8 __attribute__((ext_vector_type(8)));
typedef __bf16 bf16x2 __attribute__((ext_vector_type(2)));
typedef short s16x4 __attribute__((ext_vector_type(4)));
typedef short s16x8 __attribute__((ext_vector_type(8)));
typedef float f32x4 __attribute__((ext_vector_type(4)));
typedef float f32x16 __attribute__((ext_vector_type(16)));
typedef float f32x2 __attribute__((ext_vector_type(2)));
typedef unsigned u32x4 __attribute__((ext_vector_type(4)));
typedef unsigned u32x2 __attribute__((ext_vector_type(2)));
typedef int i32x4 __attribute__((ext_vector_type(4)));

#define LOADER_SLEEP 1
#define SPREAD 0
#define AUX_A 0
#define AUX_B 0
#define WIN 2
#define ST_AUX 17
#define PUSH_T0 150
#define PUSH_DT 70
#define POLL_SLEEP 4
#define ACQ 0
#define QAHEAD 4
#define OUT_AUX 0
#define PF 0
#define TLBW 0
#define ROT 1
#define POLL_T0 400
#define POLL_SCOPE __HIP_MEMORY_SCOPE_AGENT
#define MF32 0

namespace {

constexpr int NR = 8;
constexpr int MLOC = 512;
constexpr int KCH = 512;
constexpr int NTOT = 16384;
constexpr int NSLOT = 6;
constexpr int A_SLOT = 8192;
constexpr int B_SLOT = 16384;
constexpr int SLOT = A_SLOT + B_SLOT;
constexpr int CTL_OFF = NSLOT * SLOT;
constexpr int LDS_BYTES = CTL_OFF + 256;
constexpr int NSTEP = 128;
constexpr int NWG = 256;

constexpr unsigned FLAG_OFF = 0;
constexpr unsigned TMO_OFF = 2048;
constexpr unsigned CTR_OFF = 4096;
constexpr unsigned DATA_OFF = 65536;
constexpr unsigned PAR_BYTES = 4u << 20;
constexpr unsigned WS_NEED = DATA_OFF + 2 * PAR_BYTES;
constexpr unsigned long long DEADLINE = 50000000ull;

struct Params {
    const void* x;
    const void* w;
    void* out;
    char* ws[NR];
    int rank;
};

__device__ __forceinline__ __amdgpu_buffer_rsrc_t mk_rsrc(const void* p, unsigned bytes) {
    return __builtin_amdgcn_make_buffer_rsrc((void*)p, (short)0, (int)bytes, 0x00020000);
}

#define DMA(...) __builtin_amdgcn_raw_ptr_buffer_load_lds(__VA_ARGS__)
#define DMA_A(...) DMA(__VA_ARGS__)
#define DMA_B(...) DMA(__VA_ARGS__)
#define WAIT_VM(n) asm volatile("s_waitcnt vmcnt(" #n ")" ::: "memory")
#define LAG 3
#define WAIT_LAG() WAIT_VM(24)
#define WAIT_LGKM0() asm volatile("s_waitcnt lgkmcnt(0)" ::: "memory")

__device__ __forceinline__ unsigned pack2(float a, float b) {
    f32x2 v = {a, b};
    bf16x2 r = __builtin_convertvector(v, bf16x2);
    return __builtin_bit_cast(unsigned, r);
}

__global__ __launch_bounds__(512, 2) void a2a_gemm_kernel(Params p) {
    extern __shared__ __attribute__((aligned(16))) char smem[];
    LAS char* L = (LAS char*)smem;
    volatile LAS unsigned* ctl = (volatile LAS unsigned*)(L + CTL_OFF);

    const int tid = threadIdx.x;
    const int wave = __builtin_amdgcn_readfirstlane(tid >> 6);
    const int lane = tid & 63;
    const int b = blockIdx.x;
    const int idx = b >> 3;
    const int nb = (b & 7) * 8 + (idx >> 2);
    const int mb = idx & 3;
    const int rank = p.rank;
#define ROTB(i) (((i) & ~3) | (((i) + mb) & 3))

#define STAMP(i) do {} while (0)
    STAMP(0);
#define TR(J, i) do {} while (0)
    if (tid < 64) ctl[tid] = 0u;
    __syncthreads();

    if (wave < 4) {
        const int wn = wave;
        u32x4 fl;
        auto wait_full = [&](unsigned need) {
            for (;;) {
                u32x4 f = *(volatile LAS u32x4*)(L + CTL_OFF);
                unsigned mn = min(min(f[0], f[1]), f[2]);
                if (__builtin_amdgcn_readfirstlane(mn) >= need) break;
            }
            asm volatile("" ::: "memory");
        };
        bf16x8 xa[2][8], wb[2][4];
        const int g = lane >> 4, r15 = lane & 15, q = (lane >> 2) & 3, pp = lane & 3;
        const unsigned aoff = (unsigned)(r15 * 64 + ((g ^ ((-(r15 >> 2)) & 3)) * 16));
        unsigned boff[4];
#pragma unroll
        for (int t = 0; t < 4; ++t)
            boff[t] = (unsigned)(A_SLOT + (8 * g + q) * 512 + (((((wn & 1) * 4 + t) ^ (q | ((g & 1) << 2)))) << 5) +
                                 ((wn >> 1) << 8) + 8 * pp);
        f32x4 acc[8][4];
#pragma unroll
        for (int m = 0; m < 8; ++m)
#pragma unroll
            for (int t = 0; t < 4; ++t) acc[m][t] = (f32x4){0.f, 0.f, 0.f, 0.f};
        auto rd_w = [&](unsigned sb, int t) -> bf16x8 {
            s16x4 lo = __builtin_amdgcn_ds_read_tr16_b64_v4i16((LAS s16x4*)(L + sb + boff[t]));
            s16x4 hi = __builtin_amdgcn_ds_read_tr16_b64_v4i16((LAS s16x4*)(L + sb + boff[t] + 2048));
            s16x8 v = __builtin_shufflevector(lo, hi, 0, 1, 2, 3, 4, 5, 6, 7);
            return __builtin_bit_cast(bf16x8, v);
        };
        auto rd_x = [&](unsigned sb, int m) -> bf16x8 {
            return *(const LAS bf16x8*)(L + sb + aoff + m * 1024);
        };
#define SB_() __builtin_amdgcn_sched_barrier(0)
#define READ_ALL(U, SBASE, PIN) do { \
            wb[U][0] = rd_w(SBASE, 0); PIN; \
            _Pragma("unroll") for (int m = 0; m < 8; ++m) { xa[U][m] = rd_x(SBASE, m); PIN; } \
            _Pragma("unroll") for (int t = 1; t < 4; ++t) { wb[U][t] = rd_w(SBASE, t); PIN; } \
            fl = *(volatile LAS u32x4*)(L + CTL_OFF); PIN; } while (0)
#define MMA(U, I) acc[(I) & 7][(I) >> 3] = __builtin_amdgcn_mfma_f32_16x16x32_bf16(wb[U][(I) >> 3], xa[U][(I) & 7], acc[(I) & 7][(I) >> 3], 0, 0, 0)
#define NTAIL16 5
        constexpr int NMMA = 32, NTAIL = NTAIL16;

        wait_full(1);
        STAMP(1);
        READ_ALL(0, 0u, SB_());
        WAIT_LGKM0();
        ctl[4 + wn] = 1u;
        wait_full(2);

        unsigned sbn = SLOT;
        for (int J = 0; J < NSTEP; J += 2) {
#pragma unroll
            for (int u = 0; u < 2; ++u) {
                const int Jc = J + u;
                READ_ALL(u ^ 1, sbn, (void)0);
#pragma unroll
                for (int i = 0; i < NMMA - NTAIL; ++i) MMA(u, i);
                __builtin_amdgcn_sched_group_barrier(0x008, 1, 0);
#pragma unroll
                for (int i = 0; i < 17; ++i) {
                    __builtin_amdgcn_sched_group_barrier(0x100, 1, 0);
                    __builtin_amdgcn_sched_group_barrier(0x008, 1, 0);
                }
                __builtin_amdgcn_sched_group_barrier(0x008, NMMA - NTAIL - 18, 0);
                __builtin_amdgcn_sched_barrier(0);
                WAIT_LGKM0();
                TR(Jc, 0);
                ctl[4 + wn] = (unsigned)(Jc + 2);
                sbn += SLOT;
                if (sbn == NSLOT * SLOT) sbn = 0;
                asm volatile("" : "+s"(sbn));
                const unsigned mn = __builtin_amdgcn_readfirstlane(min(min(fl[0], fl[1]), fl[2]));
                const unsigned need = (unsigned)min(Jc + 3, NSTEP);
#pragma unroll
                for (int i = NMMA - NTAIL; i < NMMA; ++i) MMA(u, i);
                __builtin_amdgcn_sched_group_barrier(0x002, 1, 0);
                __builtin_amdgcn_sched_group_barrier(0x008, 1, 0);
                __builtin_amdgcn_sched_group_barrier(0x002, 1, 0);
                __builtin_amdgcn_sched_group_barrier(0x008, 1, 0);
                __builtin_amdgcn_sched_group_barrier(0x002, 2, 0);
                __builtin_amdgcn_sched_group_barrier(0x200, 1, 0);
                __builtin_amdgcn_sched_group_barrier(0x008, NTAIL - 2, 0);
                if (__builtin_expect(mn < need, 0)) wait_full(need);
                TR(Jc, 1);
                asm volatile("" ::: "memory");
            }
        }

        WAIT_LGKM0();
        STAMP(2);
        const unsigned bad = ctl[8];
        const __amdgpu_buffer_rsrc_t ro = mk_rsrc(p.out, (unsigned)MLOC * NTOT * 2u);
        const unsigned obase = (unsigned)(((mb * 128 + r15) * NTOT + nb * 256 + wn * 64 + 16 * (g & 1) + 8 * (g >> 1)) * 2);
#pragma unroll
        for (int m = 0; m < 8; ++m) {
#pragma unroll
            for (int tp = 0; tp < 2; ++tp) {
                unsigned a0 = pack2(acc[m][2 * tp][0], acc[m][2 * tp][1]);
                unsigned a1 = pack2(acc[m][2 * tp][2], acc[m][2 * tp][3]);
                unsigned b0 = pack2(acc[m][2 * tp + 1][0], acc[m][2 * tp + 1][1]);
                unsigned b1 = pack2(acc[m][2 * tp + 1][2], acc[m][2 * tp + 1][3]);
                auto s0 = __builtin_amdgcn_permlane16_swap(a0, b0, false, false);
                auto s1 = __builtin_amdgcn_permlane16_swap(a1, b1, false, false);
                u32x4 v = {s0[0], s1[0], s0[1], s1[1]};
                if (bad) v = (u32x4){0x7FC07FC0u, 0x7FC07FC0u, 0x7FC07FC0u, 0x7FC07FC0u};
                __builtin_amdgcn_raw_buffer_store_b128(v, ro, (int)obase, (m * 16 * NTOT + tp * 32) * 2, OUT_AUX);
            }
        }
    } else if (wave < 7) {
        const int ld = wave - 4;
        volatile LAS int* fullw = (volatile LAS int*)(L + CTL_OFF) + ld;
        i32x4 epre = {0, 0, 0, 0};
        int pub = 0;
        int J = 0;
        unsigned sb = 0;
        auto rd_freed = [&]() -> int {
            i32x4 e = *(volatile LAS i32x4*)(L + CTL_OFF + 16);
            return __builtin_amdgcn_readfirstlane(min(min(e[0], e[1]), min(e[2], e[3])));
        };
        auto pre = [&]() {
            const int need = J - (NSLOT - 1);
            int freed = __builtin_amdgcn_readfirstlane(min(min(epre[0], epre[1]), min(epre[2], epre[3])));
            if (__builtin_expect(freed < need, 0)) {
                if (pub < J - 1) { WAIT_VM(8); pub = J - 1; *fullw = pub; }
                freed = rd_freed();
                if (freed < need) {
                    if (pub < J) { WAIT_VM(0); pub = J; *fullw = pub; }
                    while ((freed = rd_freed()) < need) __builtin_amdgcn_s_sleep(LOADER_SLEEP);
                }
            }
            asm volatile("" ::: "memory");
        };
        auto post = [&]() {
            epre = *(volatile LAS i32x4*)(L + CTL_OFF + 16);
            WAIT_VM(16);
            pub = max(pub, J - 1);
            *fullw = pub;
            ++J;
            sb += SLOT;
            if (sb == NSLOT * SLOT) sb = 0;
        };
        unsigned qpre = 0;
        auto q_prefetch = [&](int t) { qpre = *(volatile LAS unsigned char*)(L + CTL_OFF + 64 + t); };
        auto q_get = [&](int t) -> unsigned {
            unsigned c = __builtin_amdgcn_readfirstlane(qpre);
            if (__builtin_expect(c == 0u, 0)) {
                if (pub < J) { WAIT_VM(0); pub = J; *fullw = pub; }
                while ((c = __builtin_amdgcn_readfirstlane((unsigned)*(volatile LAS unsigned char*)(L + CTL_OFF + 64 + t))) == 0u)
                    __builtin_amdgcn_s_sleep(1);
            }
            asm volatile("" ::: "memory");
            return c;
        };
        if (ld == 0) {
            const __amdgpu_buffer_rsrc_t rx = mk_rsrc(p.x, 4096u * KCH * 2u);
            char* wsme = p.ws[rank];
            const __amdgpu_buffer_rsrc_t rw = mk_rsrc(wsme, WS_NEED);
            const unsigned callv = __hip_atomic_load((const unsigned*)(wsme + CTR_OFF + b * 128), __ATOMIC_RELAXED, __HIP_MEMORY_SCOPE_AGENT);
            const unsigned vloc = (unsigned)((lane >> 2) * 1024 + (((lane & 3) ^ ((-(lane >> 4)) & 3)) * 16));
            const unsigned vrem = (unsigned)(lane * 16);
            const unsigned call = __builtin_amdgcn_readfirstlane(callv) + 1u;
            const unsigned so0 = (unsigned)((rank * 512 + mb * 128) * 1024);
#pragma unroll 1
            for (int i8 = 0; i8 < 8; ++i8) {
                const unsigned so = so0 + (unsigned)ROTB(i8) * 128u;
#pragma unroll
                for (int hf = 0; hf < 2; ++hf) {
                    pre();
#pragma unroll
                    for (int i = 0; i < 8; ++i)
                        DMA_A(rx, (LAS void*)(L + sb + i * 1024), 16, (int)vloc, (int)(so + hf * 64 + i * 16 * 1024), 0, AUX_A);
                    post();
                }
            }
            const unsigned dbase = DATA_OFF + (call & 1u) * PAR_BYTES + (unsigned)(mb * 16 * 8192);
            q_prefetch(0);
#pragma unroll 1
            for (int t = 0; t < 56; ++t) {
                const unsigned c = q_get(t);
                const unsigned sr = dbase + ((c >> 3) & 7u) * (64u * 8192u) + (c & 7u) * 16384u;
#pragma unroll
                for (int hf = 0; hf < 2; ++hf) {
                    pre();
#pragma unroll
                    for (int i = 0; i < 8; ++i)
                        DMA_A(rw, (LAS void*)(L + sb + i * 1024), 16, (int)vrem, (int)(sr + hf * 8192 + i * 1024), 0, AUX_A);
                    if (hf == 1) q_prefetch(t + 1);
                    post();
                }
            }
        } else {
            const int bl = ld - 1;
            const __amdgpu_buffer_rsrc_t rwm = mk_rsrc(p.w, 4096u * NTOT * 2u);
            const int h = lane >> 5;
            unsigned vo[4];
#pragma unroll
            for (int c = 0; c < 4; ++c)
                vo[c] = (unsigned)(h * 32768 + (((lane & 31) ^ ((2 * c + h) << 1)) * 16));
            const unsigned wbase = (unsigned)(16 * bl * 32768 + nb * 512);
            const unsigned lo = (unsigned)(A_SLOT + bl * 8192);
            auto issue = [&](unsigned so) {
#pragma unroll
                for (int i = 0; i < 8; ++i)
                    DMA_B(rwm, (LAS void*)(L + sb + lo + i * 1024), 16, (int)vo[(i & 1) + 2 * ((i >> 2) & 1)],
                          (int)(so + i * 2 * 32768), 0, AUX_B);
            };
#pragma unroll 1
            for (int i8 = 0; i8 < 8; ++i8) {
                const unsigned so = wbase + (unsigned)(rank * 512 + ROTB(i8) * 64) * 32768u;
#pragma unroll
                for (int hf = 0; hf < 2; ++hf) {
                    pre();
                    issue(so + hf * 32u * 32768u);
                    post();
                }
            }
            q_prefetch(0);
#pragma unroll 1
            for (int t = 0; t < 56; ++t) {
                const unsigned c = q_get(t);
                const unsigned sr = wbase + (((c >> 3) & 7u) * 512u + (c & 7u) * 64u) * 32768u;
#pragma unroll
                for (int hf = 0; hf < 2; ++hf) {
                    pre();
                    issue(sr + hf * 32u * 32768u);
                    if (hf == 1) q_prefetch(t + 1);
                    post();
                }
            }
        }
        WAIT_VM(8);  *fullw = NSTEP - 1;
        WAIT_VM(0);  *fullw = NSTEP;
    } else {
        const unsigned long long tk0 = __builtin_amdgcn_s_memrealtime();
        char* wsme = p.ws[rank];
        unsigned* ctr = (unsigned*)(wsme + CTR_OFF + b * 128);
        const unsigned callv = __hip_atomic_load(ctr, __ATOMIC_RELAXED, __HIP_MEMORY_SCOPE_AGENT);
        const int pi = b % 7, un = b / 7, jjp = (un >> 2) & 7, mbd = un & 3;
        const int dst = (rank + 1 + pi) & 7;
        char* wsd = p.ws[dst];
        const __amdgpu_buffer_rsrc_t rx = mk_rsrc(p.x, 4096u * KCH * 2u);
        const __amdgpu_buffer_rsrc_t rd = mk_rsrc(wsd, WS_NEED);
        const unsigned vs = (unsigned)((lane >> 2) * 1024 + (((lane & 3) ^ ((-(lane >> 4)) & 3)) * 16));
        const unsigned ss = (unsigned)((dst * 512 + mbd * 128) * 1024 + jjp * 128);
        u32x4 v[16];
        bool todo = b < 224;
        if (todo) {
#pragma unroll
            for (int j = 0; j < 16; ++j)
                v[j] = __builtin_amdgcn_raw_buffer_load_b128(rx, (int)vs, (int)(ss + (j & 7) * 16384 + (j >> 3) * 64), 0);
        }
        __builtin_amdgcn_sched_barrier(0);
        WAIT_VM(0);
        const unsigned call = __builtin_amdgcn_readfirstlane(callv) + 1u;
        const unsigned long long due = tk0 + (unsigned long long)(PUSH_T0 + un * PUSH_DT);
        const int cl = lane < 56 ? ROTB(lane) : 0;
        const int srcl = (rank + 1 + cl % 7) & 7, jjl = cl / 7;
        const unsigned codel = 0x80u | (unsigned)(srcl << 3) | (unsigned)jjl;
        const unsigned* fptr = (const unsigned*)(wsme + FLAG_OFF + srcl * 128 + mb * 32 + jjl * 4);
        unsigned* tmo = (unsigned*)(wsme + TMO_OFF);
        const unsigned long long want = (1ull << 56) - 1ull;
        unsigned long long have = 0ull, disp = 0ull;
        int qn = 0;
        for (;;) {
            if (todo && __builtin_amdgcn_s_memrealtime() >= due) {
                const unsigned sd = DATA_OFF + (call & 1u) * PAR_BYTES + (unsigned)(((rank * 4 + mbd) * 16 + 2 * jjp) * 8192);
                unsigned* flag = (unsigned*)(wsd + FLAG_OFF + rank * 128 + mbd * 32 + jjp * 4);
#pragma unroll
                for (int j = 0; j < 16; ++j)
                    __builtin_amdgcn_raw_buffer_store_b128(v[j], rd, lane * 16, (int)(sd + j * 1024), ST_AUX);
                WAIT_VM(0);
                if (lane == 0) __hip_atomic_store(flag, call, __ATOMIC_RELAXED, __HIP_MEMORY_SCOPE_SYSTEM);
                todo = false;
            }
            int limit = 56;
            if (PF || have != want) limit = min(56, max(0, ((int)__builtin_amdgcn_readfirstlane(ctl[0]) - 16) >> 1) + QAHEAD);
            if (have != want && qn < limit && (have & ~disp) == 0ull && __builtin_amdgcn_s_memrealtime() - tk0 >= POLL_T0) {
                const unsigned fv = __hip_atomic_load(fptr, __ATOMIC_RELAXED, POLL_SCOPE);
                unsigned long long now = __ballot(fv >= call) & want;
                const unsigned long long el = __builtin_amdgcn_s_memrealtime() - tk0;
                if (now != want && el > 100000ull &&
                    (el > DEADLINE ||
                     __builtin_amdgcn_readfirstlane(__hip_atomic_load(tmo, __ATOMIC_RELAXED, __HIP_MEMORY_SCOPE_AGENT)) == call)) {
                    __hip_atomic_store(tmo, call, __ATOMIC_RELAXED, __HIP_MEMORY_SCOPE_AGENT);
                    ctl[8] = 1u;
                    now = want;
                }
                have = now;
                if (!PF && have == want) limit = 56;
            }
            {
                unsigned long long avail = have & ~disp;
                while (qn < limit && avail) {
                    const int c = __builtin_ctzll(avail);
                    avail &= avail - 1ull;
                    disp |= 1ull << c;
                    const unsigned code = (unsigned)__builtin_amdgcn_readlane(codel, c);
                    *(volatile LAS unsigned char*)(L + CTL_OFF + 64 + qn) = (unsigned char)code;
                    ++qn;
                }
            }
            if (qn == 56 && !todo) break;
            __builtin_amdgcn_s_sleep(POLL_SLEEP);
        }
        for (;;) {
            u32x4 e = *(volatile LAS u32x4*)(L + CTL_OFF + 16);
            if (__builtin_amdgcn_readfirstlane(e[0]) >= 1u) break;
            __builtin_amdgcn_s_sleep(8);
        }
        if (lane == 0) __hip_atomic_store(ctr, call, __ATOMIC_RELAXED, __HIP_MEMORY_SCOPE_AGENT);
    }
}

}

extern "C" void kernel_launch(int rank, int nranks, ncclComm_t comm,
                              hipStream_t stream,
                              void* const* d_in, const int* in_sizes, int n_in,
                              void* d_out, int out_size,
                              const int* in_dtypes, int out_dtype,
                              void* const* d_out_all,
                              void* const* d_ws_all, size_t ws_bytes) {
    (void)comm; (void)in_sizes; (void)n_in; (void)out_size; (void)in_dtypes; (void)out_dtype; (void)d_out_all;
    if (nranks != NR || (unsigned)rank >= (unsigned)NR || ws_bytes < WS_NEED) {
        fprintf(stderr, "unexpected configuration\n");
        return;
    }
    Params p;
    p.x = d_in[0];
    p.w = d_in[1];
    p.out = d_out;
    for (int i = 0; i < NR; ++i) p.ws[i] = (char*)d_ws_all[i];
    p.rank = rank;
    HIP_CHECK(hipFuncSetAttribute((const void*)a2a_gemm_kernel, hipFuncAttributeMaxDynamicSharedMemorySize, LDS_BYTES));
    hipLaunchKernelGGL(a2a_gemm_kernel, dim3(NWG), dim3(512), LDS_BYTES, stream, p);
    HIP_CHECK(hipGetLastError());
}
```

```cpp
#include <hip/hip_runtime.h>
#include <rccl/rccl.h>
#include <cstdio>

#define HIP_CHECK(expr) do { hipError_t _e = (expr); if (_e != hipSuccess) { \
    fprintf(stderr, "HIP error %s at %s:%d\n", hipGetErrorString(_e), __FILE__, __LINE__); \
    return; } } while (0)

#define LAS __attribute__((address_space(3)))
typedef __bf16 bf16x8 __attribute__((ext_vector_type(8)));
typedef __bf16 bf16x2 __attribute__((ext_vector_type(2)));
typedef short s16x4 __attribute__((ext_vector_type(4)));
typedef short s16x8 __attribute__((ext_vector_type(8)));
typedef float f32x4 __attribute__((ext_vector_type(4)));
typedef float f32x2 __attribute__((ext_vector_type(2)));
typedef unsigned u32x4 __attribute__((ext_vector_type(4)));
typedef int i32x4 __attribute__((ext_vector_type(4)));

namespace {

constexpr int NR = 8;
constexpr int MLOC = 512;
constexpr int KCH = 512;
constexpr int NTOT = 16384;
constexpr int NSLOT = 6;
constexpr int A_SLOT = 8192;
constexpr int B_SLOT = 16384;
constexpr int SLOT = A_SLOT + B_SLOT;
constexpr int CTL_OFF = NSLOT * SLOT;
constexpr int LDS_BYTES = CTL_OFF + 128;
constexpr int NSTEP = 128;
constexpr int NWG = 256;

constexpr unsigned FLAG_OFF = 0;
constexpr unsigned TMO_OFF = 2048;
constexpr unsigned CTR_OFF = 4096;
constexpr unsigned DATA_OFF = 65536;
constexpr unsigned PAR_BYTES = 4u << 20;
constexpr unsigned WS_NEED = DATA_OFF + 2 * PAR_BYTES;

constexpr unsigned long long DEADLINE = 50000000ull;
constexpr int PUSH_T0 = 150, PUSH_DT = 70;
constexpr unsigned long long POLL_T0 = 400;
constexpr int QAHEAD = 4;

struct Params {
    const void* x;
    const void* w;
    void* out;
    char* ws[NR];
    int rank;
};

__device__ __forceinline__ __amdgpu_buffer_rsrc_t mk_rsrc(const void* p, unsigned bytes) {
    return __builtin_amdgcn_make_buffer_rsrc((void*)p, (short)0, (int)bytes, 0x00020000);
}

#define WAIT_VM(n) asm volatile("s_waitcnt vmcnt(" #n ")" ::: "memory")
#define WAIT_LGKM0() asm volatile("s_waitcnt lgkmcnt(0)" ::: "memory")
#define DMA16(rsrc, ldsoff, voff, soff) \
    __builtin_amdgcn_raw_ptr_buffer_load_lds(rsrc, (LAS void*)(L + (ldsoff)), 16, (int)(voff), (int)(soff), 0, 0)
#define START_BARRIER() do { WAIT_LGKM0(); __builtin_amdgcn_s_barrier(); asm volatile("" ::: "memory"); } while (0)

__device__ __forceinline__ unsigned pack2(float a, float b) {
    f32x2 v = {a, b};
    bf16x2 r = __builtin_convertvector(v, bf16x2);
    return __builtin_bit_cast(unsigned, r);
}

__global__ __launch_bounds__(512, 2) void a2a_gemm_kernel(Params p) {
    extern __shared__ __attribute__((aligned(16))) char smem[];
    LAS char* L = (LAS char*)smem;
    volatile LAS unsigned* ctl = (volatile LAS unsigned*)(L + CTL_OFF);

    const int tid = threadIdx.x;
    const int wave = __builtin_amdgcn_readfirstlane(tid >> 6);
    const int lane = tid & 63;
    const int b = blockIdx.x;
    const int idx = b >> 3;
    const int nb = (b & 7) * 8 + (idx >> 2);
    const int mb = idx & 3;
    const int rank = p.rank;

    if (tid < 32) ctl[tid] = 0u;

    if (wave < 4) {
        START_BARRIER();
        const int wn = wave;
        const int g = lane >> 4, r15 = lane & 15, q = (lane >> 2) & 3, pp = lane & 3;
        const unsigned aoff = (unsigned)(r15 * 64 + ((g ^ ((-(r15 >> 2)) & 3)) * 16));
        unsigned boff[4];
#pragma unroll
        for (int t = 0; t < 4; ++t)
            boff[t] = (unsigned)(A_SLOT + (8 * g + q) * 512 + (((((wn & 1) * 4 + t) ^ (q | ((g & 1) << 2)))) << 5) +
                                 ((wn >> 1) << 8) + 8 * pp);
        f32x4 acc[8][4];
#pragma unroll
        for (int m = 0; m < 8; ++m)
#pragma unroll
            for (int t = 0; t < 4; ++t) acc[m][t] = (f32x4){0.f, 0.f, 0.f, 0.f};
        bf16x8 xa[2][8], wb[2][4];
        u32x4 fl;

        auto wait_full = [&](unsigned need) {
            for (;;) {
                u32x4 f = *(volatile LAS u32x4*)(L + CTL_OFF);
                unsigned mn = min(min(f[0], f[1]), f[2]);
                if (__builtin_amdgcn_readfirstlane(mn) >= need) break;
            }
            asm volatile("" ::: "memory");
        };
        auto rd_w = [&](unsigned sb, int t) -> bf16x8 {
            s16x4 lo = __builtin_amdgcn_ds_read_tr16_b64_v4i16((LAS s16x4*)(L + sb + boff[t]));
            s16x4 hi = __builtin_amdgcn_ds_read_tr16_b64_v4i16((LAS s16x4*)(L + sb + boff[t] + 2048));
            s16x8 v = __builtin_shufflevector(lo, hi, 0, 1, 2, 3, 4, 5, 6, 7);
            return __builtin_bit_cast(bf16x8, v);
        };
        auto rd_x = [&](unsigned sb, int m) -> bf16x8 {
            return *(const LAS bf16x8*)(L + sb + aoff + m * 1024);
        };
#define READ_ALL(U, SBASE, PIN) do { \
            wb[U][0] = rd_w(SBASE, 0); PIN; \
            _Pragma("unroll") for (int m = 0; m < 8; ++m) { xa[U][m] = rd_x(SBASE, m); PIN; } \
            _Pragma("unroll") for (int t = 1; t < 4; ++t) { wb[U][t] = rd_w(SBASE, t); PIN; } \
            fl = *(volatile LAS u32x4*)(L + CTL_OFF); PIN; } while (0)
#define MM_(I) ((((I) >> 3) & 1) ? 7 - ((I) & 7) : ((I) & 7))
#define MMA(U, I) acc[MM_(I)][(I) >> 3] = __builtin_amdgcn_mfma_f32_16x16x32_bf16(wb[U][(I) >> 3], xa[U][MM_(I)], acc[MM_(I)][(I) >> 3], 0, 0, 0)
        constexpr int NMMA = 32, NTAIL = 5;

        wait_full(1);
        READ_ALL(0, 0u, __builtin_amdgcn_sched_barrier(0));
        WAIT_LGKM0();
        ctl[4 + wn] = 1u;
        wait_full(2);

        unsigned sbn = SLOT;
        for (int J = 0; J < NSTEP; J += 2) {
#pragma unroll
            for (int u = 0; u < 2; ++u) {
                const int Jc = J + u;
                READ_ALL(u ^ 1, sbn, (void)0);
#pragma unroll
                for (int i = 0; i < NMMA - NTAIL; ++i) MMA(u, i);
                __builtin_amdgcn_sched_group_barrier(0x008, 1, 0);
#pragma unroll
                for (int i = 0; i < 17; ++i) {
                    __builtin_amdgcn_sched_group_barrier(0x100, 1, 0);
                    __builtin_amdgcn_sched_group_barrier(0x008, 1, 0);
                }
                __builtin_amdgcn_sched_group_barrier(0x008, NMMA - NTAIL - 18, 0);
                __builtin_amdgcn_sched_barrier(0);
                WAIT_LGKM0();
                ctl[4 + wn] = (unsigned)(Jc + 2);
                sbn += SLOT;
                if (sbn == NSLOT * SLOT) sbn = 0;
                asm volatile("" : "+s"(sbn));
                const unsigned mn = __builtin_amdgcn_readfirstlane(min(min(fl[0], fl[1]), fl[2]));
                const unsigned need = (unsigned)min(Jc + 3, NSTEP);
#pragma unroll
                for (int i = NMMA - NTAIL; i < NMMA; ++i) MMA(u, i);
                __builtin_amdgcn_sched_group_barrier(0x002, 1, 0);
                __builtin_amdgcn_sched_group_barrier(0x008, 1, 0);
                __builtin_amdgcn_sched_group_barrier(0x002, 1, 0);
                __builtin_amdgcn_sched_group_barrier(0x008, 1, 0);
                __builtin_amdgcn_sched_group_barrier(0x002, 2, 0);
                __builtin_amdgcn_sched_group_barrier(0x200, 1, 0);
                __builtin_amdgcn_sched_group_barrier(0x008, NTAIL - 2, 0);
                if (__builtin_expect(mn < need, 0)) wait_full(need);
                asm volatile("" ::: "memory");
            }
        }

        WAIT_LGKM0();
        const unsigned bad = ctl[8];
        const __amdgpu_buffer_rsrc_t ro = mk_rsrc(p.out, (unsigned)MLOC * NTOT * 2u);
        const unsigned obase = (unsigned)(((mb * 128 + r15) * NTOT + nb * 256 + wn * 64 + 16 * (g & 1) + 8 * (g >> 1)) * 2);
        if (__builtin_expect(__builtin_amdgcn_readfirstlane(bad) != 0u, 0)) {
            const float qnan = __builtin_bit_cast(float, 0x7FC00000u);
#pragma unroll
            for (int m = 0; m < 8; ++m)
#pragma unroll
                for (int t = 0; t < 4; ++t) acc[m][t] = (f32x4){qnan, qnan, qnan, qnan};
        }
#pragma unroll
        for (int m = 0; m < 8; ++m) {
#pragma unroll
            for (int tp = 0; tp < 2; ++tp) {
                unsigned a0 = pack2(acc[m][2 * tp][0], acc[m][2 * tp][1]);
                unsigned a1 = pack2(acc[m][2 * tp][2], acc[m][2 * tp][3]);
                unsigned b0 = pack2(acc[m][2 * tp + 1][0], acc[m][2 * tp + 1][1]);
                unsigned b1 = pack2(acc[m][2 * tp + 1][2], acc[m][2 * tp + 1][3]);
                auto s0 = __builtin_amdgcn_permlane16_swap(a0, b0, false, false);
                auto s1 = __builtin_amdgcn_permlane16_swap(a1, b1, false, false);
                u32x4 v = {s0[0], s1[0], s0[1], s1[1]};
                __builtin_amdgcn_raw_buffer_store_b128(v, ro, (int)obase, (m * 16 * NTOT + tp * 32) * 2, 0);
            }
        }
    } else if (wave < 7) {
        const int ld = wave - 4;
        volatile LAS int* fullw = (volatile LAS int*)(L + CTL_OFF) + ld;
        i32x4 epre = {0, 0, 0, 0};
        int pub = 0;
        int J = 0;
        unsigned sb = 0;
        auto rd_freed = [&]() -> int {
            i32x4 e = *(volatile LAS i32x4*)(L + CTL_OFF + 16);
            return __builtin_amdgcn_readfirstlane(min(min(e[0], e[1]), min(e[2], e[3])));
        };
        auto try_publish = [&](int issued) {
            const unsigned st = __builtin_amdgcn_s_getreg(7 | (31 << 11));
            const int out = (int)(((st & 15u) | (((st >> 22) & 3u) << 4)) + 7u) >> 3;
            const int landed = issued - out;
            if (landed > pub) {
                switch (out) {
                    case 0: WAIT_VM(0); break;
                    case 1: WAIT_VM(8); break;
                    case 2: WAIT_VM(16); break;
                    case 3: WAIT_VM(24); break;
                    case 4: WAIT_VM(32); break;
                    default: WAIT_VM(40); break;
                }
                pub = landed;
                *fullw = pub;
            }
        };
        auto pre = [&]() {
            const int need = J - (NSLOT - 1);
            int freed = __builtin_amdgcn_readfirstlane(min(min(epre[0], epre[1]), min(epre[2], epre[3])));
            while (__builtin_expect(freed < need, 0)) {
                try_publish(J);
                __builtin_amdgcn_s_sleep(1);
                freed = rd_freed();
            }
            asm volatile("" ::: "memory");
        };
        auto post = [&]() {
            epre = *(volatile LAS i32x4*)(L + CTL_OFF + 16);
            ++J;
            if (J > NSLOT) { WAIT_VM(16); pub = max(pub, J - 2); *fullw = pub; } else try_publish(J);
            sb += SLOT;
            if (sb == NSLOT * SLOT) sb = 0;
        };
        unsigned qpre = 0;
        auto q_prefetch = [&](int t) { qpre = *(volatile LAS unsigned char*)(L + CTL_OFF + 64 + t); };
        auto q_get = [&](int t) -> unsigned {
            unsigned c = __builtin_amdgcn_readfirstlane(qpre);
            if (__builtin_expect(c == 0u, 0)) {
                while ((c = __builtin_amdgcn_readfirstlane((unsigned)*(volatile LAS unsigned char*)(L + CTL_OFF + 64 + t))) == 0u) {
                    try_publish(J);
                    __builtin_amdgcn_s_sleep(1);
                }
            }
            asm volatile("" ::: "memory");
            return c;
        };
        if (ld == 0) {
            const __amdgpu_buffer_rsrc_t rx = mk_rsrc(p.x, 4096u * KCH * 2u);
            const __amdgpu_buffer_rsrc_t rw = mk_rsrc(p.ws[rank], WS_NEED);
            const unsigned vloc = (unsigned)((lane >> 2) * 1024 + (((lane & 3) ^ ((-(lane >> 4)) & 3)) * 16));
            const unsigned vrem = (unsigned)(lane * 16);
            const unsigned so0 = (unsigned)((rank * 512 + mb * 128) * 1024);
#pragma unroll 1
            for (int i8 = 0; i8 < 8; ++i8) {
#pragma unroll
                for (int hf = 0; hf < 2; ++hf) {
                    pre();
#pragma unroll
                    for (int i = 0; i < 8; ++i) DMA16(rx, sb + i * 1024, vloc, so0 + i8 * 128 + hf * 64 + i * 16 * 1024);
                    if (hf == 0 && i8 == 0) START_BARRIER();
                    post();
                }
            }
            const unsigned dbase = DATA_OFF + (unsigned)(mb * 16 * 8192);
            q_prefetch(0);
#pragma unroll 1
            for (int t = 0; t < 56; ++t) {
                const unsigned c = q_get(t);
                const unsigned sr = dbase + ((c >> 3) & 15u) * (64u * 8192u) + (c & 7u) * 16384u;
#pragma unroll
                for (int hf = 0; hf < 2; ++hf) {
                    pre();
#pragma unroll
                    for (int i = 0; i < 8; ++i) DMA16(rw, sb + i * 1024, vrem, sr + hf * 8192 + i * 1024);
                    if (hf == 1) q_prefetch(t + 1);
                    post();
                }
            }
        } else {
            const int bl = ld - 1;
            const __amdgpu_buffer_rsrc_t rwm = mk_rsrc(p.w, 4096u * NTOT * 2u);
            const int h = lane >> 5;
            unsigned vo[4];
#pragma unroll
            for (int c = 0; c < 4; ++c) vo[c] = (unsigned)(h * 32768 + (((lane & 31) ^ ((2 * c + h) << 1)) * 16));
            const unsigned wbase = (unsigned)(16 * bl * 32768 + nb * 512);
            const unsigned lo = (unsigned)(A_SLOT + bl * 8192);
            auto issue = [&](unsigned so) {
#pragma unroll
                for (int i = 0; i < 8; ++i)
                    DMA16(rwm, sb + lo + i * 1024, vo[(i & 1) + 2 * ((i >> 2) & 1)], so + i * 2 * 32768);
            };
#pragma unroll 1
            for (int i8 = 0; i8 < 8; ++i8) {
                const unsigned so = wbase + (unsigned)(rank * 512 + i8 * 64) * 32768u;
#pragma unroll
                for (int hf = 0; hf < 2; ++hf) {
                    pre();
                    issue(so + hf * 32u * 32768u);
                    if (hf == 0 && i8 == 0) START_BARRIER();
                    post();
                }
            }
            q_prefetch(0);
#pragma unroll 1
            for (int t = 0; t < 56; ++t) {
                const unsigned c = q_get(t);
                const unsigned sr = wbase + (((c >> 3) & 7u) * 512u + (c & 7u) * 64u) * 32768u;
#pragma unroll
                for (int hf = 0; hf < 2; ++hf) {
                    pre();
                    issue(sr + hf * 32u * 32768u);
                    if (hf == 1) q_prefetch(t + 1);
                    post();
                }
            }
        }
        WAIT_VM(8);  *fullw = NSTEP - 1;
        WAIT_VM(0);  *fullw = NSTEP;
    } else {
        START_BARRIER();
        const unsigned long long tk0 = __builtin_amdgcn_s_memrealtime();
        char* wsme = p.ws[rank];
        unsigned* ctr = (unsigned*)(wsme + CTR_OFF + b * 128);
        const unsigned callv = __hip_atomic_load(ctr, __ATOMIC_RELAXED, __HIP_MEMORY_SCOPE_AGENT);
        const int pi = b % 7, un = b / 7, jjp = (un >> 2) & 7, mbd = un & 3;
        const int dst = (rank + 1 + pi) & 7;
        char* wsd = p.ws[dst];
        const __amdgpu_buffer_rsrc_t rx = mk_rsrc(p.x, 4096u * KCH * 2u);
        const __amdgpu_buffer_rsrc_t rd = mk_rsrc(wsd, WS_NEED);
        const unsigned vs = (unsigned)((lane >> 2) * 1024 + (((lane & 3) ^ ((-(lane >> 4)) & 3)) * 16));
        const unsigned ss = (unsigned)((dst * 512 + mbd * 128) * 1024 + jjp * 128);
        u32x4 v[16];
        bool todo = b < 224;
        if (todo) {
#pragma unroll
            for (int j = 0; j < 16; ++j)
                v[j] = __builtin_amdgcn_raw_buffer_load_b128(rx, (int)vs, (int)(ss + (j & 7) * 16384 + (j >> 3) * 64), 0);
        }
        __builtin_amdgcn_sched_barrier(0);
        WAIT_VM(0);
        const unsigned call = __builtin_amdgcn_readfirstlane(callv) + 1u;
        if (lane == 0) __hip_atomic_store(ctr, call, __ATOMIC_RELAXED, __HIP_MEMORY_SCOPE_AGENT);
        const unsigned long long due = tk0 + (unsigned long long)(PUSH_T0 + un * PUSH_DT);
        const int cl = lane < 56 ? lane : 0;
        const int srcl = (rank + 1 + cl % 7) & 7, jjl = cl / 7;
        const unsigned codel = 0x80u | ((call & 1u) << 6) | (unsigned)(srcl << 3) | (unsigned)jjl;
        const unsigned* fptr = (const unsigned*)(wsme + FLAG_OFF + srcl * 128 + mb * 32 + jjl * 4);
        unsigned* tmo = (unsigned*)(wsme + TMO_OFF);
        const unsigned long long want = (1ull << 56) - 1ull;
        unsigned long long have = 0ull, disp = 0ull;
        int qn = 0;
        for (;;) {
            if (todo && __builtin_amdgcn_s_memrealtime() >= due) {
                const unsigned sd = DATA_OFF + (call & 1u) * PAR_BYTES + (unsigned)(((rank * 4 + mbd) * 16 + 2 * jjp) * 8192);
                unsigned* flag = (unsigned*)(wsd + FLAG_OFF + rank * 128 + mbd * 32 + jjp * 4);
#pragma unroll
                for (int j = 0; j < 16; ++j)
                    __builtin_amdgcn_raw_buffer_store_b128(v[j], rd, lane * 16, (int)(sd + j * 1024), 17  );
                WAIT_VM(0);
                if (lane == 0) __hip_atomic_store(flag, call, __ATOMIC_RELAXED, __HIP_MEMORY_SCOPE_SYSTEM);
                todo = false;
            }
            int limit = 56;
            if (have != want) limit = min(56, max(0, ((int)__builtin_amdgcn_readfirstlane(ctl[0]) - 16) >> 1) + QAHEAD);
            if (have != want && qn < limit && (have & ~disp) == 0ull && __builtin_amdgcn_s_memrealtime() - tk0 >= POLL_T0) {
                const unsigned fv = __hip_atomic_load(fptr, __ATOMIC_RELAXED, __HIP_MEMORY_SCOPE_SYSTEM);
                unsigned long long now = __ballot(fv >= call) & want;
                const unsigned long long el = __builtin_amdgcn_s_memrealtime() - tk0;
                if (now != want && el > 100000ull &&
                    (el > DEADLINE ||
                     __builtin_amdgcn_readfirstlane(__hip_atomic_load(tmo, __ATOMIC_RELAXED, __HIP_MEMORY_SCOPE_AGENT)) == call)) {
                    __hip_atomic_store(tmo, call, __ATOMIC_RELAXED, __HIP_MEMORY_SCOPE_AGENT);
                    ctl[8] = 1u;
                    now = want;
                }
                have = now;
                if (have == want) limit = 56;
            }
            unsigned long long avail = have & ~disp;
            while (qn < limit && avail) {
                const int c = __builtin_ctzll(avail);
                avail &= avail - 1ull;
                disp |= 1ull << c;
                *(volatile LAS unsigned char*)(L + CTL_OFF + 64 + qn) = (unsigned char)__builtin_amdgcn_readlane(codel, c);
                ++qn;
            }
            if (qn == 56 && !todo) break;
            __builtin_amdgcn_s_sleep(4);
        }
    }
}

}

extern "C" void kernel_launch(int rank, int nranks, ncclComm_t comm,
                              hipStream_t stream,
                              void* const* d_in, const int* in_sizes, int n_in,
                              void* d_out, int out_size,
                              const int* in_dtypes, int out_dtype,
                              void* const* d_out_all,
                              void* const* d_ws_all, size_t ws_bytes) {
    (void)comm; (void)in_sizes; (void)n_in; (void)out_size; (void)in_dtypes; (void)out_dtype; (void)d_out_all;
    if (nranks != NR || (unsigned)rank >= (unsigned)NR || ws_bytes < WS_NEED) {
        fprintf(stderr, "unexpected configuration\n");
        return;
    }
    Params p;
    p.x = d_in[0];
    p.w = d_in[1];
    p.out = d_out;
    for (int i = 0; i < NR; ++i) p.ws[i] = (char*)d_ws_all[i];
    p.rank = rank;
    HIP_CHECK(hipFuncSetAttribute((const void*)a2a_gemm_kernel, hipFuncAttributeMaxDynamicSharedMemorySize, LDS_BYTES));
    hipLaunchKernelGGL(a2a_gemm_kernel, dim3(NWG), dim3(512), LDS_BYTES, stream, p);
    HIP_CHECK(hipGetLastError());
}
```

```cpp
#include <hip/hip_runtime.h>
#include <rccl/rccl.h>
#include <cstdio>

#define HIP_CHECK(expr) do { hipError_t _e = (expr); if (_e != hipSuccess) { \
    fprintf(stderr, "HIP error %s at %s:%d\n", hipGetErrorString(_e), __FILE__, __LINE__); \
    return; } } while (0)

#define LAS __attribute__((address_space(3)))
typedef __bf16 bf16x8 __attribute__((ext_vector_type(8)));
typedef __bf16 bf16x2 __attribute__((ext_vector_type(2)));
typedef short s16x4 __attribute__((ext_vector_type(4)));
typedef short s16x8 __attribute__((ext_vector_type(8)));
typedef float f32x4 __attribute__((ext_vector_type(4)));
typedef float f32x2 __attribute__((ext_vector_type(2)));
typedef unsigned u32x4 __attribute__((ext_vector_type(4)));
typedef int i32x4 __attribute__((ext_vector_type(4)));

namespace {

constexpr int NR = 8;
constexpr int MLOC = 512;
constexpr int KCH = 512;
constexpr int NTOT = 16384;
constexpr int NSLOT = 6;
constexpr int A_SLOT = 8192;
constexpr int B_SLOT = 16384;
constexpr int SLOT = A_SLOT + B_SLOT;
constexpr int CTL_OFF = NSLOT * SLOT;
constexpr int LDS_BYTES = CTL_OFF + 128;
constexpr int NSTEP = 128;
constexpr int NWG = 256;

constexpr unsigned FLAG_OFF = 0;
constexpr unsigned TMO_OFF = 2048;
constexpr unsigned CTR_OFF = 4096;
constexpr unsigned DATA_OFF = 65536;
constexpr unsigned PAR_BYTES = 4u << 20;
constexpr unsigned WS_NEED = DATA_OFF + 2 * PAR_BYTES;

constexpr unsigned long long DEADLINE = 50000000ull;
constexpr int PUSH_T0 = 150, PUSH_DT = 70;
constexpr int PUSH_DT_LATE = 30, LATE_CNT = 49;
constexpr unsigned long long POLL_T0 = 400;
constexpr int QAHEAD = 4;

struct Params {
    const void* x;
    const void* w;
    void* out;
    char* ws[NR];
    int rank;
};

__device__ __forceinline__ __amdgpu_buffer_rsrc_t mk_rsrc(const void* p, unsigned bytes) {
    return __builtin_amdgcn_make_buffer_rsrc((void*)p, (short)0, (int)bytes, 0x00020000);
}

#define WAIT_VM(n) asm volatile("s_waitcnt vmcnt(" #n ")" ::: "memory")
#define WAIT_LGKM0() asm volatile("s_waitcnt lgkmcnt(0)" ::: "memory")
#define DMA16(rsrc, ldsoff, voff, soff) \
    __builtin_amdgcn_raw_ptr_buffer_load_lds(rsrc, (LAS void*)(L + (ldsoff)), 16, (int)(voff), (int)(soff), 0, 0)
#define DMA16_SYS(rsrc, ldsoff, voff, soff) \
    __builtin_amdgcn_raw_ptr_buffer_load_lds(rsrc, (LAS void*)(L + (ldsoff)), 16, (int)(voff), (int)(soff), 0, 17)
#define START_BARRIER() do { WAIT_LGKM0(); __builtin_amdgcn_s_barrier(); asm volatile("" ::: "memory"); } while (0)

__device__ __forceinline__ unsigned pack2(float a, float b) {
    f32x2 v = {a, b};
    bf16x2 r = __builtin_convertvector(v, bf16x2);
    return __builtin_bit_cast(unsigned, r);
}

__global__ __launch_bounds__(512, 2) void a2a_gemm_kernel(Params p) {
    extern __shared__ __attribute__((aligned(16))) char smem[];
    LAS char* L = (LAS char*)smem;
    volatile LAS unsigned* ctl = (volatile LAS unsigned*)(L + CTL_OFF);

    const int tid = threadIdx.x;
    const int wave = __builtin_amdgcn_readfirstlane(tid >> 6);
    const int lane = tid & 63;
    const int b = blockIdx.x;
    const int idx = b >> 3;
    const int nb = (b & 7) * 8 + (idx >> 2);
    const int mb = idx & 3;
    const int rank = p.rank;

    if (tid < 32) ctl[tid] = 0u;

    if (wave < 4) {
        START_BARRIER();
        const int wn = wave;
        const int g = lane >> 4, r15 = lane & 15, q = (lane >> 2) & 3, pp = lane & 3;
        const unsigned aoff = (unsigned)(r15 * 64 + ((g ^ ((-(r15 >> 2)) & 3)) * 16));
        unsigned boff[4];
#pragma unroll
        for (int t = 0; t < 4; ++t)
            boff[t] = (unsigned)(A_SLOT + (8 * g + q) * 512 + (((((wn & 1) * 4 + t) ^ (q | ((g & 1) << 2)))) << 5) +
                                 ((wn >> 1) << 8) + 8 * pp);
        f32x4 acc[8][4];
#pragma unroll
        for (int m = 0; m < 8; ++m)
#pragma unroll
            for (int t = 0; t < 4; ++t) acc[m][t] = (f32x4){0.f, 0.f, 0.f, 0.f};
        bf16x8 xa[2][8], wb[2][4];
        u32x4 fl;

        auto wait_full = [&](unsigned need) {
            for (;;) {
                u32x4 f = *(volatile LAS u32x4*)(L + CTL_OFF);
                unsigned mn = min(min(f[0], f[1]), f[2]);
                if (__builtin_amdgcn_readfirstlane(mn) >= need) break;
            }
            asm volatile("" ::: "memory");
        };
        auto rd_w = [&](unsigned sb, int t) -> bf16x8 {
            s16x4 lo = __builtin_amdgcn_ds_read_tr16_b64_v4i16((LAS s16x4*)(L + sb + boff[t]));
            s16x4 hi = __builtin_amdgcn_ds_read_tr16_b64_v4i16((LAS s16x4*)(L + sb + boff[t] + 2048));
            s16x8 v = __builtin_shufflevector(lo, hi, 0, 1, 2, 3, 4, 5, 6, 7);
            return __builtin_bit_cast(bf16x8, v);
        };
        auto rd_x = [&](unsigned sb, int m) -> bf16x8 {
            return *(const LAS bf16x8*)(L + sb + aoff + m * 1024);
        };
#define READ_ALL(U, SBASE, PIN) do { \
            wb[U][0] = rd_w(SBASE, 0); PIN; \
            _Pragma("unroll") for (int m = 0; m < 8; ++m) { xa[U][m] = rd_x(SBASE, m); PIN; } \
            _Pragma("unroll") for (int t = 1; t < 4; ++t) { wb[U][t] = rd_w(SBASE, t); PIN; } \
            fl = *(volatile LAS u32x4*)(L + CTL_OFF); PIN; } while (0)
#define MM_(I) ((((I) >> 3) & 1) ? 7 - ((I) & 7) : ((I) & 7))
#define MMA(U, I) acc[MM_(I)][(I) >> 3] = __builtin_amdgcn_mfma_f32_16x16x32_bf16(wb[U][(I) >> 3], xa[U][MM_(I)], acc[MM_(I)][(I) >> 3], 0, 0, 0)
        constexpr int NMMA = 32, NTAIL = 5;

        wait_full(1);
        READ_ALL(0, 0u, __builtin_amdgcn_sched_barrier(0));
        WAIT_LGKM0();
        ctl[4 + wn] = 1u;
        wait_full(2);

        unsigned sbn = SLOT;
        for (int J = 0; J < NSTEP; J += 2) {
#pragma unroll
            for (int u = 0; u < 2; ++u) {
                const int Jc = J + u;
                READ_ALL(u ^ 1, sbn, (void)0);
#pragma unroll
                for (int i = 0; i < NMMA - NTAIL; ++i) MMA(u, i);
                __builtin_amdgcn_sched_group_barrier(0x008, 1, 0);
#pragma unroll
                for (int i = 0; i < 17; ++i) {
                    __builtin_amdgcn_sched_group_barrier(0x100, 1, 0);
                    __builtin_amdgcn_sched_group_barrier(0x008, 1, 0);
                }
                __builtin_amdgcn_sched_group_barrier(0x008, NMMA - NTAIL - 18, 0);
                __builtin_amdgcn_sched_barrier(0);
                WAIT_LGKM0();
                ctl[4 + wn] = (unsigned)(Jc + 2);
                sbn += SLOT;
                if (sbn == NSLOT * SLOT) sbn = 0;
                asm volatile("" : "+s"(sbn));
                const unsigned mn = __builtin_amdgcn_readfirstlane(min(min(fl[0], fl[1]), fl[2]));
                const unsigned need = (unsigned)min(Jc + 3, NSTEP);
#pragma unroll
                for (int i = NMMA - NTAIL; i < NMMA; ++i) MMA(u, i);
                __builtin_amdgcn_sched_group_barrier(0x002, 1, 0);
                __builtin_amdgcn_sched_group_barrier(0x008, 1, 0);
                __builtin_amdgcn_sched_group_barrier(0x002, 1, 0);
                __builtin_amdgcn_sched_group_barrier(0x008, 1, 0);
                __builtin_amdgcn_sched_group_barrier(0x002, 2, 0);
                __builtin_amdgcn_sched_group_barrier(0x200, 1, 0);
                __builtin_amdgcn_sched_group_barrier(0x008, NTAIL - 2, 0);
                if (__builtin_expect(mn < need, 0)) wait_full(need);
                asm volatile("" ::: "memory");
            }
        }

        WAIT_LGKM0();
        const unsigned bad = ctl[8];
        const __amdgpu_buffer_rsrc_t ro = mk_rsrc(p.out, (unsigned)MLOC * NTOT * 2u);
        const unsigned obase = (unsigned)(((mb * 128 + r15) * NTOT + nb * 256 + wn * 64 + 16 * (g & 1) + 8 * (g >> 1)) * 2);
        if (__builtin_expect(__builtin_amdgcn_readfirstlane(bad) != 0u, 0)) {
            const float qnan = __builtin_bit_cast(float, 0x7FC00000u);
#pragma unroll
            for (int m = 0; m < 8; ++m)
#pragma unroll
                for (int t = 0; t < 4; ++t) acc[m][t] = (f32x4){qnan, qnan, qnan, qnan};
        }
#pragma unroll
        for (int m = 0; m < 8; ++m) {
#pragma unroll
            for (int tp = 0; tp < 2; ++tp) {
                unsigned a0 = pack2(acc[m][2 * tp][0], acc[m][2 * tp][1]);
                unsigned a1 = pack2(acc[m][2 * tp][2], acc[m][2 * tp][3]);
                unsigned b0 = pack2(acc[m][2 * tp + 1][0], acc[m][2 * tp + 1][1]);
                unsigned b1 = pack2(acc[m][2 * tp + 1][2], acc[m][2 * tp + 1][3]);
                auto s0 = __builtin_amdgcn_permlane16_swap(a0, b0, false, false);
                auto s1 = __builtin_amdgcn_permlane16_swap(a1, b1, false, false);
                u32x4 v = {s0[0], s1[0], s0[1], s1[1]};
                __builtin_amdgcn_raw_buffer_store_b128(v, ro, (int)obase, (m * 16 * NTOT + tp * 32) * 2, 0);
            }
        }
    } else if (wave < 7) {
        const int ld = wave - 4;
        volatile LAS int* fullw = (volatile LAS int*)(L + CTL_OFF) + ld;
        i32x4 epre = {0, 0, 0, 0};
        int pub = 0;
        int J = 0;
        unsigned sb = 0;
        auto rd_freed = [&]() -> int {
            i32x4 e = *(volatile LAS i32x4*)(L + CTL_OFF + 16);
            return __builtin_amdgcn_readfirstlane(min(min(e[0], e[1]), min(e[2], e[3])));
        };
        auto try_publish = [&](int issued) {
            const unsigned st = __builtin_amdgcn_s_getreg(7 | (31 << 11));
            const int out = (int)(((st & 15u) | (((st >> 22) & 3u) << 4)) + 7u) >> 3;
            const int landed = issued - out;
            if (landed > pub) {
                switch (out) {
                    case 0: WAIT_VM(0); break;
                    case 1: WAIT_VM(8); break;
                    case 2: WAIT_VM(16); break;
                    case 3: WAIT_VM(24); break;
                    case 4: WAIT_VM(32); break;
                    default: WAIT_VM(40); break;
                }
                pub = landed;
                *fullw = pub;
            }
        };
        auto pre = [&]() {
            const int need = J - (NSLOT - 1);
            int freed = __builtin_amdgcn_readfirstlane(min(min(epre[0], epre[1]), min(epre[2], epre[3])));
            while (__builtin_expect(freed < need, 0)) {
                try_publish(J);
                __builtin_amdgcn_s_sleep(1);
                freed = rd_freed();
            }
            asm volatile("" ::: "memory");
        };
        auto post = [&]() {
            epre = *(volatile LAS i32x4*)(L + CTL_OFF + 16);
            ++J;
            if (J > NSLOT) { WAIT_VM(16); pub = max(pub, J - 2); *fullw = pub; } else try_publish(J);
            sb += SLOT;
            if (sb == NSLOT * SLOT) sb = 0;
        };
        unsigned qpre = 0;
        auto q_prefetch = [&](int t) { qpre = *(volatile LAS unsigned char*)(L + CTL_OFF + 64 + t); };
        auto q_get = [&](int t) -> unsigned {
            unsigned c = __builtin_amdgcn_readfirstlane(qpre);
            if (__builtin_expect(c == 0u, 0)) {
                while ((c = __builtin_amdgcn_readfirstlane((unsigned)*(volatile LAS unsigned char*)(L + CTL_OFF + 64 + t))) == 0u) {
                    try_publish(J);
                    __builtin_amdgcn_s_sleep(1);
                }
            }
            asm volatile("" ::: "memory");
            return c;
        };
        if (ld == 0) {
            const __amdgpu_buffer_rsrc_t rx = mk_rsrc(p.x, 4096u * KCH * 2u);
            const __amdgpu_buffer_rsrc_t rw = mk_rsrc(p.ws[rank], WS_NEED);
            const unsigned vloc = (unsigned)((lane >> 2) * 1024 + (((lane & 3) ^ ((-(lane >> 4)) & 3)) * 16));
            const unsigned vrem = (unsigned)(lane * 16);
            const unsigned so0 = (unsigned)((rank * 512 + mb * 128) * 1024);
#pragma unroll 1
            for (int i8 = 0; i8 < 8; ++i8) {
#pragma unroll
                for (int hf = 0; hf < 2; ++hf) {
                    pre();
#pragma unroll
                    for (int i = 0; i < 8; ++i) DMA16(rx, sb + i * 1024, vloc, so0 + i8 * 128 + hf * 64 + i * 16 * 1024);
                    if (hf == 0 && i8 == 0) START_BARRIER();
                    post();
                }
            }
            const unsigned dbase = DATA_OFF + (unsigned)(mb * 16 * 8192);
            q_prefetch(0);
#pragma unroll 1
            for (int t = 0; t < 56; ++t) {
                const unsigned c = q_get(t);
                const unsigned sr = dbase + ((c >> 3) & 15u) * (64u * 8192u) + (c & 7u) * 16384u;
#pragma unroll
                for (int hf = 0; hf < 2; ++hf) {
                    pre();
#pragma unroll
                    for (int i = 0; i < 8; ++i) DMA16_SYS(rw, sb + i * 1024, vrem, sr + hf * 8192 + i * 1024);
                    if (hf == 1) q_prefetch(t + 1);
                    post();
                }
            }
        } else {
            const int bl = ld - 1;
            const __amdgpu_buffer_rsrc_t rwm = mk_rsrc(p.w, 4096u * NTOT * 2u);
            const int h = lane >> 5;
            unsigned vo[4];
#pragma unroll
            for (int c = 0; c < 4; ++c) vo[c] = (unsigned)(h * 32768 + (((lane & 31) ^ ((2 * c + h) << 1)) * 16));
            const unsigned wbase = (unsigned)(16 * bl * 32768 + nb * 512);
            const unsigned lo = (unsigned)(A_SLOT + bl * 8192);
            auto issue = [&](unsigned so) {
#pragma unroll
                for (int i = 0; i < 8; ++i)
                    DMA16(rwm, sb + lo + i * 1024, vo[(i & 1) + 2 * ((i >> 2) & 1)], so + i * 2 * 32768);
            };
#pragma unroll 1
            for (int i8 = 0; i8 < 8; ++i8) {
                const unsigned so = wbase + (unsigned)(rank * 512 + i8 * 64) * 32768u;
#pragma unroll
                for (int hf = 0; hf < 2; ++hf) {
                    pre();
                    issue(so + hf * 32u * 32768u);
                    if (hf == 0 && i8 == 0) START_BARRIER();
                    post();
                }
            }
            q_prefetch(0);
#pragma unroll 1
            for (int t = 0; t < 56; ++t) {
                const unsigned c = q_get(t);
                const unsigned sr = wbase + (((c >> 3) & 7u) * 512u + (c & 7u) * 64u) * 32768u;
#pragma unroll
                for (int hf = 0; hf < 2; ++hf) {
                    pre();
                    issue(sr + hf * 32u * 32768u);
                    if (hf == 1) q_prefetch(t + 1);
                    post();
                }
            }
        }
        WAIT_VM(8);  *fullw = NSTEP - 1;
        WAIT_VM(0);  *fullw = NSTEP;
    } else {
        START_BARRIER();
        const unsigned long long tk0 = __builtin_amdgcn_s_memrealtime();
        char* wsme = p.ws[rank];
        unsigned* ctr = (unsigned*)(wsme + CTR_OFF + b * 128);
        const unsigned callv = __hip_atomic_load(ctr, __ATOMIC_RELAXED, __HIP_MEMORY_SCOPE_AGENT);
        const int pi = b % 7, un = b / 7, jjp = (un >> 2) & 7, mbd = un & 3;
        const int dst = (rank + 1 + pi) & 7;
        char* wsd = p.ws[dst];
        const __amdgpu_buffer_rsrc_t rx = mk_rsrc(p.x, 4096u * KCH * 2u);
        const __amdgpu_buffer_rsrc_t rd = mk_rsrc(wsd, WS_NEED);
        const unsigned vs = (unsigned)((lane >> 2) * 1024 + (((lane & 3) ^ ((-(lane >> 4)) & 3)) * 16));
        const unsigned ss = (unsigned)((dst * 512 + mbd * 128) * 1024 + jjp * 128);
        const int cl = lane < 56 ? lane : 0;
        const int srcl = (rank + 1 + cl % 7) & 7, jjl = cl / 7;
        const unsigned* fptr = (const unsigned*)(wsme + FLAG_OFF + srcl * 128 + mb * 32 + jjl * 4);
        const unsigned fv0 = __hip_atomic_load(fptr, __ATOMIC_RELAXED, __HIP_MEMORY_SCOPE_SYSTEM);
        u32x4 v[16];
        bool todo = b < 224;
        if (todo) {
#pragma unroll
            for (int j = 0; j < 16; ++j)
                v[j] = __builtin_amdgcn_raw_buffer_load_b128(rx, (int)vs, (int)(ss + (j & 7) * 16384 + (j >> 3) * 64), 0);
        }
        __builtin_amdgcn_sched_barrier(0);
        WAIT_VM(0);
        const unsigned call = __builtin_amdgcn_readfirstlane(callv) + 1u;
        if (lane == 0) __hip_atomic_store(ctr, call, __ATOMIC_RELAXED, __HIP_MEMORY_SCOPE_AGENT);
        const unsigned codel = 0x80u | ((call & 1u) << 6) | (unsigned)(srcl << 3) | (unsigned)jjl;
        unsigned* tmo = (unsigned*)(wsme + TMO_OFF);
        const unsigned long long want = (1ull << 56) - 1ull;
        unsigned long long have = __ballot(fv0 >= call) & want, disp = 0ull;
        const bool late = __builtin_popcountll(have) >= LATE_CNT;
        const unsigned long long due = tk0 + (unsigned long long)(late ? un * PUSH_DT_LATE : PUSH_T0 + un * PUSH_DT);
        int qn = 0;
        for (;;) {
            if (todo && __builtin_amdgcn_s_memrealtime() >= due) {
                const unsigned sd = DATA_OFF + (call & 1u) * PAR_BYTES + (unsigned)(((rank * 4 + mbd) * 16 + 2 * jjp) * 8192);
                unsigned* flag = (unsigned*)(wsd + FLAG_OFF + rank * 128 + mbd * 32 + jjp * 4);
#pragma unroll
                for (int j = 0; j < 16; ++j)
                    __builtin_amdgcn_raw_buffer_store_b128(v[j], rd, lane * 16, (int)(sd + j * 1024), 17  );
                WAIT_VM(0);
                if (lane == 0) __hip_atomic_store(flag, call, __ATOMIC_RELAXED, __HIP_MEMORY_SCOPE_SYSTEM);
                todo = false;
            }
            int limit = 56;
            if (have != want) limit = min(56, max(0, ((int)__builtin_amdgcn_readfirstlane(ctl[0]) - 16) >> 1) + QAHEAD);
            if (have != want && qn < limit && (have & ~disp) == 0ull && __builtin_amdgcn_s_memrealtime() - tk0 >= POLL_T0) {
                const unsigned fv = __hip_atomic_load(fptr, __ATOMIC_RELAXED, __HIP_MEMORY_SCOPE_SYSTEM);
                unsigned long long now = __ballot(fv >= call) & want;
                const unsigned long long el = __builtin_amdgcn_s_memrealtime() - tk0;
                if (now != want && el > 100000ull &&
                    (el > DEADLINE ||
                     __builtin_amdgcn_readfirstlane(__hip_atomic_load(tmo, __ATOMIC_RELAXED, __HIP_MEMORY_SCOPE_AGENT)) == call)) {
                    __hip_atomic_store(tmo, call, __ATOMIC_RELAXED, __HIP_MEMORY_SCOPE_AGENT);
                    ctl[8] = 1u;
                    now = want;
                }
                have = now;
                if (have == want) limit = 56;
            }
            unsigned long long avail = have & ~disp;
            while (qn < limit && avail) {
                const int c = __builtin_ctzll(avail);
                avail &= avail - 1ull;
                disp |= 1ull << c;
                *(volatile LAS unsigned char*)(L + CTL_OFF + 64 + qn) = (unsigned char)__builtin_amdgcn_readlane(codel, c);
                ++qn;
            }
            if (qn == 56 && !todo) break;
            __builtin_amdgcn_s_sleep(4);
        }
    }
}

}

extern "C" void kernel_launch(int rank, int nranks, ncclComm_t comm,
                              hipStream_t stream,
                              void* const* d_in, const int* in_sizes, int n_in,
                              void* d_out, int out_size,
                              const int* in_dtypes, int out_dtype,
                              void* const* d_out_all,
                              void* const* d_ws_all, size_t ws_bytes) {
    (void)comm; (void)in_sizes; (void)n_in; (void)out_size; (void)in_dtypes; (void)out_dtype; (void)d_out_all;
    if (nranks != NR || (unsigned)rank >= (unsigned)NR || ws_bytes < WS_NEED) {
        fprintf(stderr, "unexpected configuration\n");
        return;
    }
    Params p;
    p.x = d_in[0];
    p.w = d_in[1];
    p.out = d_out;
    for (int i = 0; i < NR; ++i) p.ws[i] = (char*)d_ws_all[i];
    p.rank = rank;
    HIP_CHECK(hipFuncSetAttribute((const void*)a2a_gemm_kernel, hipFuncAttributeMaxDynamicSharedMemorySize, LDS_BYTES));
    hipLaunchKernelGGL(a2a_gemm_kernel, dim3(NWG), dim3(512), LDS_BYTES, stream, p);
    HIP_CHECK(hipGetLastError());
}
```
